```python
import numpy as np

kernel_name = "gpu_gemm_ar_m4096_k4096_n16384_e4m3e5m2_dqrelu"


ABSMAX_RTOL = 0.05

COMPUTE_DTYPE = "float32"
EPILOGUE = "dequant_relu"
IN_QUANT = None
OUT_QUANT = None
IN_FP8 = "e4m3e5m2"
IN_DTYPES = {"x": "e4m3", "w_mat": "e5m2",
             "scale_x": "float32", "scale_w": "float32"}
WORLD_SIZE = 8

def _snap_fp8(x, mant_bits, max_abs, min_normal):
    a = np.abs(x.astype(np.float64))
    _m, _e = np.frexp(a)
    q = np.where(a >= min_normal, np.ldexp(1.0, _e - 1 - mant_bits),
                 min_normal * 2.0 ** -mant_bits)
    snapped = np.minimum(np.round(a / q) * q, max_abs)
    return (np.sign(x) * snapped).astype(np.float32)

def setup_inputs():

    x = _snap_fp8(np.random.randn(4096, 4096).astype(np.float32), 3, 448.0, 2.0 ** -6)
    w_mat = _snap_fp8(np.random.randn(4096, 16384).astype(np.float32), 2, 57344.0, 2.0 ** -14)
    scale_x = (0.02 * (1.0 + np.random.rand(1))).astype(np.float32)
    scale_w = (0.02 * (1.0 + np.random.rand(1))).astype(np.float32)
    return {"x": x, "w_mat": w_mat, "scale_x": scale_x, "scale_w": scale_w}

def reference(x, w_mat, scale_x, scale_w):
    acc = (x @ w_mat).astype(np.float32)
    y = (acc * (scale_x * scale_w)).astype(np.float32)
    return np.maximum(y, 0.0).astype(np.float32)
```

```cpp
#include <hip/hip_runtime.h>
#include <rccl/rccl.h>
#include <cstdio>
#include <cstdint>
#include <algorithm>

#define HIP_CHECK(expr) do { hipError_t _e = (expr); if (_e != hipSuccess) { \
    fprintf(stderr, "HIP error %s at %s:%d\n", hipGetErrorString(_e), __FILE__, __LINE__); \
    return; } } while (0)

namespace fk {

#define LAS __attribute__((address_space(3)))
typedef int v8i __attribute__((ext_vector_type(8)));
typedef int v4i __attribute__((ext_vector_type(4)));
typedef unsigned v4u __attribute__((ext_vector_type(4)));
typedef float f32x4 __attribute__((ext_vector_type(4)));

constexpr int M_ = 4096, N_ = 16384, K_ = 4096, NR = 8, KSH = K_ / NR;
constexpr int NKT = K_ / 128;
constexpr unsigned BLK = 16384;
constexpr size_t XI_BYTES = (size_t)(M_ / 128) * NKT * BLK;
constexpr size_t WI_BYTES = (size_t)(N_ / 128) * NKT * BLK;
constexpr size_t BUF_BYTES = XI_BYTES + WI_BYTES;
constexpr size_t CTRL = 1u << 20;
constexpr size_t WS_NEED = CTRL + 2 * BUF_BYTES;
constexpr int NUX = M_ / 256, NUW = N_ / 256, NU = NUX + NUW;
constexpr size_t OFF_TICKET = 0, OFF_TMO = 256, OFF_FLAGS = 4096;
constexpr int GRID = 256;
#define NPUSH 16
constexpr int LDS_BYTES = 8 * (int)BLK;
constexpr unsigned long long DEADLINE = 50000000ull;

struct Params {
    const unsigned char* x;
    const unsigned char* w;
    const float* sx; const float* sw;
    float* out;
    char* ws[NR];
    int rank;
    unsigned char uorder[NU];
    unsigned short tiles[8][128];
};

__device__ __forceinline__ void store_sys16(char* p, v4u v) {
    asm volatile("global_store_dwordx4 %0, %1, off sc0 sc1\n\ts_nop 1" :: "v"(p), "v"(v) : "memory");
}

__device__ __forceinline__ void push_phase(const Params& p, LAS unsigned char* lds, unsigned call, int pw, int npw) {
    const int lane = threadIdx.x & 63, wid = __builtin_amdgcn_readfirstlane(threadIdx.x >> 6);
    LAS unsigned char* myl = lds + wid * BLK;
    const size_t bufoff = CTRL + (size_t)(call & 1u) * BUF_BYTES;
    for (int it = pw; it < NU * 64; it += npw) {
        const int dd = it & 7, s = (it >> 3) & 7, up = it >> 6;
        const int u = p.uorder[up];
        const unsigned d = (unsigned)(p.rank + 1 + dd) & 7u;
        const int h = s >> 2, ks = s & 3;
        v4u v[16];
        size_t dst;
        if (u < NUX) {
            const unsigned char* src = p.x + (size_t)(u * 256 + h * 128 + (lane & 15)) * KSH + ks * 128 + (lane >> 4) * 32;
#pragma unroll
            for (int q = 0; q < 16; ++q) v[q] = *(const v4u*)(src + (q >> 1) * 16 * KSH + (q & 1) * 16);
            dst = ((size_t)(u * 2 + h) * NKT + p.rank * 4 + ks) * BLK;
        } else {
            const int j = u - NUX, a = lane & 7, b = lane >> 3;
            const unsigned char* src = p.w + (size_t)(ks * 128 + b * 16) * N_ + j * 256 + h * 128 + a * 16;
            v4u r[16];
#pragma unroll
            for (int q = 0; q < 16; ++q) r[q] = *(const v4u*)(src + (size_t)q * N_);
            v4u o[16];
#pragma unroll
            for (int wv = 0; wv < 4; ++wv)
#pragma unroll
                for (int g = 0; g < 4; ++g) {
                    const unsigned r0 = r[4 * g + 0][wv], r1 = r[4 * g + 1][wv], r2 = r[4 * g + 2][wv], r3 = r[4 * g + 3][wv];
                    const unsigned t0 = __builtin_amdgcn_perm(r1, r0, 0x05010400u), t1 = __builtin_amdgcn_perm(r1, r0, 0x07030602u);
                    const unsigned t2 = __builtin_amdgcn_perm(r3, r2, 0x05010400u), t3 = __builtin_amdgcn_perm(r3, r2, 0x07030602u);
                    o[4 * wv + 0][g] = __builtin_amdgcn_perm(t2, t0, 0x05040100u);
                    o[4 * wv + 1][g] = __builtin_amdgcn_perm(t2, t0, 0x07060302u);
                    o[4 * wv + 2][g] = __builtin_amdgcn_perm(t3, t1, 0x05040100u);
                    o[4 * wv + 3][g] = __builtin_amdgcn_perm(t3, t1, 0x07060302u);
                }
            LAS unsigned char* wl = myl + a * 2048 + (b & 1) * 1024 + (b >> 1) * 256;
#pragma unroll
            for (int c = 0; c < 16; ++c) *(LAS v4u*)(wl + c * 16) = o[c];
            asm volatile("s_waitcnt lgkmcnt(0)" ::: "memory");
            __builtin_amdgcn_wave_barrier();
#pragma unroll
            for (int q = 0; q < 16; ++q) v[q] = *(LAS v4u*)(myl + q * 1024 + lane * 16);
            asm volatile("s_waitcnt lgkmcnt(0)" ::: "memory");
            __builtin_amdgcn_wave_barrier();
            dst = XI_BYTES + ((size_t)(j * 2 + h) * NKT + p.rank * 4 + ks) * BLK;
        }
        char* base = p.ws[d];
        char* dp = base + bufoff + dst + lane * 16;
#pragma unroll
        for (int q = 0; q < 16; ++q) store_sys16(dp + q * 1024, v[q]);
        asm volatile("s_waitcnt vmcnt(0)" ::: "memory");
        if (lane == 0)
            __hip_atomic_store((unsigned*)(base + OFF_FLAGS) + u * 64 + p.rank * 8 + s, call, __ATOMIC_RELAXED, __HIP_MEMORY_SCOPE_SYSTEM);
    }
}

__device__ __forceinline__ void wait_tile(const Params& p, unsigned call, int i, int j, LAS unsigned* badw) {
    if (threadIdx.x < 64) {
        char* ws = p.ws[p.rank];
        const unsigned* fx = (const unsigned*)(ws + OFF_FLAGS) + i * 64 + threadIdx.x;
        const unsigned* fw = (const unsigned*)(ws + OFF_FLAGS) + (NUX + j) * 64 + threadIdx.x;
        unsigned* tmo = (unsigned*)(ws + OFF_TMO);
        const unsigned long long t0 = __builtin_amdgcn_s_memrealtime();
        for (;;) {
            const unsigned a = __hip_atomic_load(fx, __ATOMIC_RELAXED, __HIP_MEMORY_SCOPE_SYSTEM);
            const unsigned b = __hip_atomic_load(fw, __ATOMIC_RELAXED, __HIP_MEMORY_SCOPE_SYSTEM);
            if (__all(a >= call && b >= call)) break;
            const unsigned tv = (unsigned)__builtin_amdgcn_readfirstlane(__hip_atomic_load(tmo, __ATOMIC_RELAXED, __HIP_MEMORY_SCOPE_AGENT));
            if (tv == call || __builtin_amdgcn_s_memrealtime() - t0 > DEADLINE) {
                if (threadIdx.x == 0) { __hip_atomic_store(tmo, call, __ATOMIC_RELAXED, __HIP_MEMORY_SCOPE_AGENT); *badw = 1u; }
                break;
            }
            __builtin_amdgcn_s_sleep(4);
        }
        __builtin_amdgcn_fence(__ATOMIC_ACQUIRE, "");
        asm volatile("s_waitcnt vmcnt(0) lgkmcnt(0)" ::: "memory");
    }
    asm volatile("" ::: "memory"); __builtin_amdgcn_s_barrier(); asm volatile("" ::: "memory");
}

__device__ __forceinline__ void gemm_phase(const Params& p, LAS unsigned char* lds, LAS unsigned* badw, unsigned call, int xcd, int r0, int rstride) {
    const int tid = threadIdx.x, wid = __builtin_amdgcn_readfirstlane(tid >> 6), lane = tid & 63, wr = wid >> 2, wc = wid & 3, fr = lane & 15, fq = lane >> 4;
    const char* XI = p.ws[p.rank] + CTRL + (size_t)(call & 1u) * BUF_BYTES;
    const char* WI = XI + XI_BYTES;
    const float scale = p.sx[0] * p.sw[0];
    constexpr size_t kstep = BLK, hstep = (size_t)NKT * BLK, tstep = 2 * hstep;
    const unsigned voff = (unsigned)tid * 16u;
    const unsigned ldsw = (unsigned)wid * 1024u;
    const int aoff = wr * 4 * 2048 + lane * 16, boff = wc * 2 * 2048 + lane * 16;
#define SA(b, h) (((b) * 2 + (h)) * (int)BLK)
#define SB(b, h) ((4 + (b) * 2 + (h)) * (int)BLK)
#define STAGE(bufoff, gbase) do { _Pragma("unroll") for (int _i = 0; _i < 2; ++_i) \
        __builtin_amdgcn_global_load_lds((const unsigned*)((const char*)(gbase) + voff + _i * 8192), (LAS unsigned*)(lds + (bufoff) + ldsw + _i * 8192), 16, 0, 0); } while (0)
#define LDFRAG(off) __builtin_shufflevector(*(const LAS v4i*)(lds + (off)), *(const LAS v4i*)(lds + (off) + 1024), 0, 1, 2, 3, 4, 5, 6, 7)
#define LDA(dst, b, h) do { _Pragma("unroll") for (int m = 0; m < 4; ++m) dst[m] = LDFRAG(SA(b, h) + aoff + m * 2048); } while (0)
#define LDB(dst, b, h) do { _Pragma("unroll") for (int n = 0; n < 2; ++n) dst[n] = LDFRAG(SB(b, h) + boff + n * 2048); } while (0)
#define MMA(ai, bj, At, Bt) do { __builtin_amdgcn_s_setprio(1); _Pragma("unroll") for (int m = 0; m < 4; ++m) _Pragma("unroll") for (int n = 0; n < 2; ++n) \
        acc[ai][bj][m][n] = __builtin_amdgcn_mfma_scale_f32_16x16x128_f8f6f4(Bt[n], At[m], acc[ai][bj][m][n], 1, 0, 0, 0, 0, 0); __builtin_amdgcn_s_setprio(0); } while (0)
#define WAIT_V(n) asm volatile("s_waitcnt vmcnt(" #n ")" ::: "memory")
#define WAIT_L(n) asm volatile("s_waitcnt lgkmcnt(" #n ")" ::: "memory")
#define BAR __builtin_amdgcn_s_barrier()
#define SCHED __builtin_amdgcn_sched_barrier(0)
    int li = r0;
    if (li >= 128) return;
    int cur = p.tiles[xcd][li];
    f32x4 acc[2][2][4][2];
#pragma unroll
    for (int a = 0; a < 2; ++a)
#pragma unroll
        for (int b = 0; b < 2; ++b)
#pragma unroll
            for (int m = 0; m < 4; ++m)
#pragma unroll
                for (int n = 0; n < 2; ++n) acc[a][b][m][n] = (f32x4){0.f, 0.f, 0.f, 0.f};
    v8i At[4], B0[2], B1[2];
    const char* cA = XI + (size_t)(cur >> 8) * tstep; const char* cB = WI + (size_t)(cur & 255) * tstep;
    wait_tile(p, call, cur >> 8, cur & 255, badw);
    STAGE(SB(0, 0), cB); STAGE(SA(0, 0), cA); STAGE(SB(0, 1), cB + hstep); STAGE(SA(0, 1), cA + hstep);
    if (wr == 1) BAR;
    WAIT_V(4); BAR;
    STAGE(SB(1, 0), cB + kstep); STAGE(SA(1, 0), cA + kstep); STAGE(SB(1, 1), cB + hstep + kstep);
    WAIT_V(6); BAR;
    for (;;) {
        const bool has_next = li + rstride < 128;
        const int nxt = has_next ? p.tiles[xcd][li + rstride] : cur;
        const char* nA = XI + (size_t)(nxt >> 8) * tstep; const char* nB = WI + (size_t)(nxt & 255) * tstep;
        for (int t = 0; t < NKT; t += 2) {
            const bool last = (t == NKT - 2);
            const char* a1 = cA + (size_t)(t + 1) * kstep;
            const char* a2 = last ? nA : cA + (size_t)(t + 2) * kstep; const char* b2 = last ? nB : cB + (size_t)(t + 2) * kstep;
            const char* a3 = a2 + kstep; const char* b3 = b2 + kstep;
            if (last && has_next) wait_tile(p, call, nxt >> 8, nxt & 255, badw);
            LDB(B0, 0, 0); SCHED; LDA(At, 0, 0); STAGE(SA(1, 1), a1 + hstep);
            WAIT_L(8); BAR; WAIT_L(0); MMA(0, 0, At, B0); BAR; SCHED;
            LDB(B1, 0, 1); STAGE(SB(0, 0), b2);
            BAR; WAIT_L(0); MMA(0, 1, At, B1); BAR;
            LDA(At, 0, 1); STAGE(SA(0, 0), a2);
            BAR; WAIT_L(0); MMA(1, 0, At, B0); BAR; SCHED;
            STAGE(SB(0, 1), b2 + hstep);
            WAIT_V(6); BAR; MMA(1, 1, At, B1); BAR;
            LDB(B0, 1, 0); SCHED; LDA(At, 1, 0); STAGE(SA(0, 1), a2 + hstep);
            WAIT_L(8); BAR; WAIT_L(0); MMA(0, 0, At, B0); BAR; SCHED;
            LDB(B1, 1, 1); STAGE(SB(1, 0), b3);
            BAR; WAIT_L(0); MMA(0, 1, At, B1); BAR;
            LDA(At, 1, 1); STAGE(SA(1, 0), a3);
            BAR; WAIT_L(0); MMA(1, 0, At, B0); BAR; SCHED;
            STAGE(SB(1, 1), b3 + hstep);
            WAIT_V(6); BAR; MMA(1, 1, At, B1); BAR;
        }
        {
            const bool bad = (*badw) != 0u;
            const float qn = __builtin_nanf("");
            float* rowp0 = p.out + (size_t)((cur >> 8) * 256 + wr * 64 + fr) * N_ + (cur & 255) * 256 + wc * 32 + 4 * fq;
#pragma unroll
            for (int ai = 0; ai < 2; ++ai)
#pragma unroll
                for (int m = 0; m < 4; ++m) {
                    float* rowp = rowp0 + (size_t)(ai * 128 + m * 16) * N_;
#pragma unroll
                    for (int bj = 0; bj < 2; ++bj)
#pragma unroll
                        for (int n = 0; n < 2; ++n) {
                            f32x4 y = acc[ai][bj][m][n] * scale;
#pragma unroll
                            for (int e = 0; e < 4; ++e) y[e] = bad ? qn : fmaxf(y[e], 0.f);
                            *(f32x4*)(rowp + bj * 128 + n * 16) = y;
                        }
                }
        }
        if (!has_next) break;
#pragma unroll
        for (int a = 0; a < 2; ++a)
#pragma unroll
            for (int b = 0; b < 2; ++b)
#pragma unroll
                for (int m = 0; m < 4; ++m)
#pragma unroll
                    for (int n = 0; n < 2; ++n) acc[a][b][m][n] = (f32x4){0.f, 0.f, 0.f, 0.f};
        cur = nxt; cA = nA; cB = nB; li += rstride;
    }
    WAIT_V(0);
    if (wr == 0) BAR;
    BAR;
#undef SA
#undef SB
#undef STAGE
#undef LDFRAG
#undef LDA
#undef LDB
#undef MMA
#undef WAIT_V
#undef WAIT_L
#undef BAR
#undef SCHED
}

__global__ __launch_bounds__(512, 2) void fused_kernel(const Params p) {
    extern __shared__ __attribute__((aligned(16))) unsigned char shm[];
    LAS unsigned char* lds = (LAS unsigned char*)shm;
    LAS unsigned* ctl = (LAS unsigned*)(lds + LDS_BYTES);
    if (threadIdx.x == 0) {
        const unsigned ticket = __hip_atomic_fetch_add((unsigned*)(p.ws[p.rank] + OFF_TICKET), 1u, __ATOMIC_RELAXED, __HIP_MEMORY_SCOPE_AGENT);
        ctl[0] = ticket / (unsigned)GRID + 1u; ctl[1] = 0u;
    }
    __syncthreads();
    const unsigned call = (unsigned)__builtin_amdgcn_readfirstlane(ctl[0]);
    const int bid = blockIdx.x;
    if (bid < NPUSH) {
        push_phase(p, lds, call, bid * 8 + (int)(threadIdx.x >> 6), NPUSH * 8);
        return;
    }
    gemm_phase(p, lds, ctl + 1, call, bid & 7, (bid - NPUSH) >> 3, (GRID - NPUSH) / 8);
}

static void build_tables(Params& p) {
    int n = 0;
    for (int k = 0; k < 4; ++k) {
        p.uorder[n++] = 2 * k; p.uorder[n++] = 8 + 2 * k; p.uorder[n++] = 2 * k + 1; p.uorder[n++] = 9 + 2 * k;
        for (int c = 0; c < 4; ++c) p.uorder[n++] = NUX + 4 * k + c;
    }
    for (int j = 16; j < NUW; ++j) p.uorder[n++] = NUX + j;
    int pos[NU];
    for (int t = 0; t < NU; ++t) pos[p.uorder[t]] = t;
    for (int x = 0; x < 8; ++x) {
        int keys[128], c = 0;
        for (int j = (x >> 1); j < NUW; j += 4)
            for (int i = (x & 1) * 8; i < (x & 1) * 8 + 8; ++i) {
                const int rdy = std::max(pos[i], pos[NUX + j]);
                keys[c++] = (rdy << 16) | (j << 8) | i;
            }
        std::sort(keys, keys + 128);
        for (int t = 0; t < 128; ++t) p.tiles[x][t] = (unsigned short)(((keys[t] & 255) << 8) | ((keys[t] >> 8) & 255));
    }
}

}

extern "C" void kernel_launch(int rank, int nranks, ncclComm_t comm,
                              hipStream_t stream,
                              void* const* d_in, const int* in_sizes, int n_in,
                              void* d_out, int out_size,
                              const int* in_dtypes, int out_dtype,
                              void* const* d_out_all,
                              void* const* d_ws_all, size_t ws_bytes) {
    using namespace fk;
    if (nranks != NR || ws_bytes < WS_NEED || (unsigned)rank >= (unsigned)NR) { fprintf(stderr, "unexpected configuration\n"); return; }
    static bool attr_done = false;
    if (!attr_done) {
        HIP_CHECK(hipFuncSetAttribute((const void*)fused_kernel, hipFuncAttributeMaxDynamicSharedMemorySize, LDS_BYTES + 64));
        attr_done = true;
    }
    Params p;
    p.x = (const unsigned char*)d_in[0]; p.w = (const unsigned char*)d_in[1];
    p.sx = (const float*)d_in[2]; p.sw = (const float*)d_in[3];
    p.out = (float*)d_out; p.rank = rank;
    for (int r = 0; r < NR; ++r) p.ws[r] = (char*)d_ws_all[r];
    build_tables(p);
    hipLaunchKernelGGL(fused_kernel, dim3(GRID), dim3(512), LDS_BYTES + 64, stream, p);
    HIP_CHECK(hipGetLastError());
}
```

```cpp
#include <hip/hip_runtime.h>
#include <rccl/rccl.h>
#include <cstdio>
#include <cstdint>
#include <algorithm>

#define HIP_CHECK(expr) do { hipError_t _e = (expr); if (_e != hipSuccess) { \
    fprintf(stderr, "HIP error %s at %s:%d\n", hipGetErrorString(_e), __FILE__, __LINE__); \
    return; } } while (0)

namespace fk {

#define LAS __attribute__((address_space(3)))
typedef int v8i __attribute__((ext_vector_type(8)));
typedef int v4i __attribute__((ext_vector_type(4)));
typedef unsigned v4u __attribute__((ext_vector_type(4)));
typedef float f32x4 __attribute__((ext_vector_type(4)));

constexpr int M_ = 4096, N_ = 16384, K_ = 4096, NR = 8, KSH = K_ / NR;
constexpr int NKT = K_ / 128;
constexpr unsigned BLK = 16384;
constexpr size_t XI_BYTES = (size_t)(M_ / 128) * NKT * BLK;
constexpr size_t WI_BYTES = (size_t)(N_ / 128) * NKT * BLK;
constexpr size_t BUF_BYTES = XI_BYTES + WI_BYTES;
constexpr size_t CTRL = 1u << 20;
constexpr size_t WS_NEED = CTRL + 2 * BUF_BYTES;
constexpr int NUX = M_ / 256, NUW = N_ / 256, NU = NUX + NUW;
constexpr size_t OFF_TICKET = 0, OFF_TMO = 256, OFF_FLAGS = 4096;
constexpr int GRID = 256;
#define ACQ 2
#define DEPTH 1
#define STFL "sc0 sc1"
#define PWV 2
#define NPUSH 32
constexpr int LDS_BYTES = 8 * (int)BLK;
constexpr unsigned long long DEADLINE = 50000000ull;

struct Params {
    const unsigned char* x;
    const unsigned char* w;
    const float* sx; const float* sw;
    float* out;
    char* ws[NR];
    int rank;
    int mode;
};

#define TS(k) do { } while (0)
#define TS0(k) do { } while (0)

__device__ __forceinline__ void item_decode(int it, int& u, int& h, int& ks) {
    h = it & 1;
    if (it < 256) { ks = it >> 6; const int r = (it >> 1) & 31; u = (r & 1) ? NUX + (r >> 1) : (r >> 1); }
    else { const int t = it - 256; ks = (t >> 5) & 3; u = NUX + 16 + (t >> 7) * 16 + ((t >> 1) & 15); }
}

__device__ __forceinline__ void push_load(const Params& p, int it, int lane, v4u (&r)[16]) {
    int u, h, ks; item_decode(it, u, h, ks);
    if (u < NUX) {
        const unsigned char* src = p.x + (size_t)(u * 256 + h * 128) * KSH + ks * 128;
        const unsigned vo = (unsigned)(lane & 15) * KSH + (unsigned)(lane >> 4) * 32u;
#pragma unroll
        for (int q = 0; q < 16; ++q)
            asm volatile("global_load_dwordx4 %0, %1, %2 offset:%3" : "=v"(r[q]) : "v"(vo), "s"(src + (q >> 1) * 16 * KSH), "n"((q & 1) * 16) : "memory");
    } else {
        const unsigned char* src = p.w + (size_t)(ks * 128) * N_ + (u - NUX) * 256 + h * 128;
        const unsigned vo = (unsigned)(lane >> 3) * 16u * N_ + (unsigned)(lane & 7) * 16u;
#pragma unroll
        for (int q = 0; q < 16; ++q)
            asm volatile("global_load_dwordx4 %0, %1, %2" : "=v"(r[q]) : "v"(vo), "s"(src + (size_t)q * N_) : "memory");
    }
}

__device__ __forceinline__ void push_phase(const Params& p, LAS unsigned char* lds, unsigned call, unsigned d, int pw, int npw) {
    const int lane = threadIdx.x & 63, wid = __builtin_amdgcn_readfirstlane(threadIdx.x >> 6);
    LAS unsigned char* myl = lds + wid * BLK;
    char* dbase = p.ws[d & 7u];
    const char* dimg = dbase + CTRL + (size_t)(call & 1u) * BUF_BYTES;
    unsigned* dflag = (unsigned*)(dbase + OFF_FLAGS);
    const unsigned lo = (unsigned)lane * 16u;
    constexpr int NITEM = NU * 8;
    v4u r[16];
    push_load(p, pw, lane, r);
    unsigned pfidx = 0;
    for (int it = pw; it < NITEM; it += npw) {
        int u, h, ks; item_decode(it, u, h, ks);
        if (it == pw) asm volatile("s_waitcnt vmcnt(0)" ::: "memory");
        else if (it == pw + npw) asm volatile("s_waitcnt vmcnt(16)" ::: "memory");
        else asm volatile("s_waitcnt vmcnt(17)" ::: "memory");
        asm volatile("" : "+v"(r[0]), "+v"(r[1]), "+v"(r[2]), "+v"(r[3]), "+v"(r[4]), "+v"(r[5]), "+v"(r[6]), "+v"(r[7]),
                          "+v"(r[8]), "+v"(r[9]), "+v"(r[10]), "+v"(r[11]), "+v"(r[12]), "+v"(r[13]), "+v"(r[14]), "+v"(r[15]));
        v4u v[16];
        if (u < NUX) {
#pragma unroll
            for (int q = 0; q < 16; ++q) v[q] = r[q];
        } else {
            const int a = lane & 7, b = lane >> 3;
            LAS unsigned char* wl = myl + a * 2048 + (b & 1) * 1024 + (b >> 1) * 256;
#pragma unroll
            for (int wv = 0; wv < 4; ++wv) {
                v4u o[4];
#pragma unroll
                for (int g = 0; g < 4; ++g) {
                    const unsigned r0 = r[4 * g + 0][wv], r1 = r[4 * g + 1][wv], r2 = r[4 * g + 2][wv], r3 = r[4 * g + 3][wv];
                    const unsigned t0 = __builtin_amdgcn_perm(r1, r0, 0x05010400u), t1 = __builtin_amdgcn_perm(r1, r0, 0x07030602u);
                    const unsigned t2 = __builtin_amdgcn_perm(r3, r2, 0x05010400u), t3 = __builtin_amdgcn_perm(r3, r2, 0x07030602u);
                    o[0][g] = __builtin_amdgcn_perm(t2, t0, 0x05040100u);
                    o[1][g] = __builtin_amdgcn_perm(t2, t0, 0x07060302u);
                    o[2][g] = __builtin_amdgcn_perm(t3, t1, 0x05040100u);
                    o[3][g] = __builtin_amdgcn_perm(t3, t1, 0x07060302u);
                }
#pragma unroll
                for (int c = 0; c < 4; ++c) *(LAS v4u*)(wl + (4 * wv + c) * 16) = o[c];
            }
            asm volatile("s_waitcnt lgkmcnt(0)" ::: "memory");
            __builtin_amdgcn_wave_barrier();
#pragma unroll
            for (int q = 0; q < 16; ++q) v[q] = *(LAS v4u*)(myl + q * 1024 + lane * 16);
            asm volatile("s_waitcnt lgkmcnt(0)" ::: "memory");
            __builtin_amdgcn_wave_barrier();
        }
        push_load(p, it + npw < NITEM ? it + npw : it, lane, r);
        const char* base = dimg + ((size_t)(u * 2 + h) * NKT + (size_t)(ks * 8 + p.rank)) * BLK;
#pragma unroll
        for (int q = 0; q < 16; ++q)
            asm volatile("global_store_dwordx4 %0, %1, %2 offset:%3 " STFL "\n\ts_nop 1" :: "v"(lo), "v"(v[q]), "s"(base + (q >> 2) * 4096), "n"((q & 3) * 1024) : "memory");
        if (it != pw) {
            if (it == pw + npw) asm volatile("s_waitcnt vmcnt(32)" ::: "memory");
            else asm volatile("s_waitcnt vmcnt(33)" ::: "memory");
            if (lane == 0) __hip_atomic_store(dflag + pfidx, call, __ATOMIC_RELAXED, __HIP_MEMORY_SCOPE_SYSTEM);
        }
        pfidx = (unsigned)(u * 64 + ks * 16 + h * 8 + p.rank);
    }
    asm volatile("s_waitcnt vmcnt(0)" ::: "memory");
    if (lane == 0) __hip_atomic_store(dflag + pfidx, call, __ATOMIC_RELAXED, __HIP_MEMORY_SCOPE_SYSTEM);
    TS(1 + wid);
}

__device__ __forceinline__ void poll_quarter(const Params& p, unsigned call, int i, int j, int q, LAS unsigned* badw) {
    const int lane = threadIdx.x & 63;
    char* ws = p.ws[p.rank];
    const unsigned* f = (const unsigned*)(ws + OFF_FLAGS) + ((lane & 16) ? (NUX + j) : i) * 64 + q * 16 + (lane & 15);
    unsigned* tmo = (unsigned*)(ws + OFF_TMO);
    const unsigned long long t0 = __builtin_amdgcn_s_memrealtime();
    for (;;) {
        const unsigned a = __hip_atomic_load(f, __ATOMIC_RELAXED, __HIP_MEMORY_SCOPE_SYSTEM);
        if (__all(a >= call)) break;
        const unsigned tv = (unsigned)__builtin_amdgcn_readfirstlane(__hip_atomic_load(tmo, __ATOMIC_RELAXED, __HIP_MEMORY_SCOPE_AGENT));
        if (tv == call || __builtin_amdgcn_s_memrealtime() - t0 > DEADLINE) {
            if (lane == 0) { __hip_atomic_store(tmo, call, __ATOMIC_RELAXED, __HIP_MEMORY_SCOPE_AGENT); *badw = 1u; }
            break;
        }
        __builtin_amdgcn_s_sleep(2);
    }
    __builtin_amdgcn_fence(__ATOMIC_ACQUIRE, "agent");
    asm volatile("s_waitcnt vmcnt(0) lgkmcnt(0)" ::: "memory");
}

__device__ __forceinline__ void gemm_phase(const Params& p, LAS unsigned char* lds, LAS unsigned* badw, unsigned call, int xcd, int r0, int rstride) {
    const int tid = threadIdx.x, wid = __builtin_amdgcn_readfirstlane(tid >> 6), lane = tid & 63, wr = wid >> 2, wc = wid & 3;
    char* wsl = p.ws[p.rank];
    const __amdgpu_buffer_rsrc_t rsI = __builtin_amdgcn_make_buffer_rsrc(wsl + CTRL + (size_t)(call & 1u) * BUF_BYTES, 0, (int)BUF_BYTES, 0x00020000);
    const __amdgpu_buffer_rsrc_t rsF = __builtin_amdgcn_make_buffer_rsrc(wsl + OFF_FLAGS, 0, NU * 256, 0x00020000);
    const __amdgpu_buffer_rsrc_t rsO = __builtin_amdgcn_make_buffer_rsrc(p.out, 0, (int)((size_t)M_ * N_ * 4), 0x00020000);
    const float scale = __builtin_bit_cast(float, __builtin_amdgcn_readfirstlane(__builtin_bit_cast(int, p.sx[0] * p.sw[0])));
    constexpr unsigned kstep = BLK, hstep = NKT * BLK, tstep = 2 * hstep;
    const unsigned voff = (unsigned)tid * 16u;
    const unsigned ldsw = (unsigned)wid * 1024u;
    unsigned abase = (unsigned)(size_t)lds + wr * 4 * 2048 + lane * 16, bbase = (unsigned)(size_t)lds + 4 * BLK + wc * 2 * 2048 + lane * 16;
    asm volatile("" : "+v"(abase), "+v"(bbase));
    const unsigned flane = (unsigned)(lane & 15) * 4u + (wid ? 0x40000000u : 0u);
#define SA(b, h) (((b) * 2 + (h)) * (int)BLK)
#define SB(b, h) ((4 + (b) * 2 + (h)) * (int)BLK)
#define STAGE(bufoff, soff) do { _Pragma("unroll") for (int _i = 0; _i < 2; ++_i) \
        __builtin_amdgcn_raw_ptr_buffer_load_lds(rsI, (LAS void*)(lds + (bufoff) + ldsw + _i * 8192), 16, voff, (soff) + _i * 8192, 0, 0); } while (0)
#define LDFRAG(base, off) __builtin_shufflevector(*(const LAS v4i*)(size_t)((base) + (off)), *(const LAS v4i*)(size_t)((base) + (off) + 1024), 0, 1, 2, 3, 4, 5, 6, 7)
#define LDA(dst, b, h) do { _Pragma("unroll") for (int m = 0; m < 4; ++m) dst[m] = LDFRAG(abase, SA(b, h) + m * 2048); } while (0)
#define LDB(dst, b, h) do { _Pragma("unroll") for (int n = 0; n < 2; ++n) dst[n] = LDFRAG(bbase, SA(b, h) + n * 2048); } while (0)
#define MMA(ai, bj, At, Bt) do { __builtin_amdgcn_s_setprio(1); _Pragma("unroll") for (int m = 0; m < 4; ++m) _Pragma("unroll") for (int n = 0; n < 2; ++n) \
        acc[ai][bj][m][n] = __builtin_amdgcn_mfma_scale_f32_16x16x128_f8f6f4(Bt[n], At[m], acc[ai][bj][m][n], 1, 0, 0, 0, 0, 0); __builtin_amdgcn_s_setprio(0); } while (0)
#define WAIT_V(n) asm volatile("s_waitcnt vmcnt(" #n ")" ::: "memory")
#define WAIT_L(n) asm volatile("s_waitcnt lgkmcnt(" #n ")" ::: "memory")
#define BAR __builtin_amdgcn_s_barrier()
#define SCHED __builtin_amdgcn_sched_barrier(0)
    int li = r0;
    if (li >= 128) return;
    int ci = (xcd & 1) * 8 + (li & 7), cj = (li >> 3) * 4 + (xcd >> 1);
    f32x4 acc[2][2][4][2];
#pragma unroll
    for (int a = 0; a < 2; ++a)
#pragma unroll
        for (int b = 0; b < 2; ++b)
#pragma unroll
            for (int m = 0; m < 4; ++m)
#pragma unroll
                for (int n = 0; n < 2; ++n) acc[a][b][m][n] = (f32x4){0.f, 0.f, 0.f, 0.f};
    v8i At[4], B0[2], B1[2];
    unsigned cA = (unsigned)ci * tstep, cB = (unsigned)(NUX + cj) * tstep;
    int tsk = 1;
    if (wid == 0) poll_quarter(p, call, ci, cj, 0, badw);
    asm volatile("" ::: "memory"); BAR; asm volatile("" ::: "memory");
    TS0(tsk); ++tsk;
    STAGE(SB(0, 0), cB); STAGE(SA(0, 0), cA); STAGE(SB(0, 1), cB + hstep); STAGE(SA(0, 1), cA + hstep);
    if (wr == 1) BAR;
    WAIT_V(4); BAR;
    STAGE(SB(1, 0), cB + kstep); STAGE(SA(1, 0), cA + kstep); STAGE(SB(1, 1), cB + hstep + kstep);
    WAIT_V(6); BAR;
    for (;;) {
        const bool has_next = li + rstride < 128;
        const int nl = has_next ? li + rstride : li;
        const int ni = (xcd & 1) * 8 + (nl & 7), nj = (nl >> 3) * 4 + (xcd >> 1);
        const unsigned nA = (unsigned)ni * tstep, nB = (unsigned)(NUX + nj) * tstep;
        unsigned long long rdy = 0;
        const unsigned so = (unsigned)ci * (256u * N_ * 4u) + (unsigned)cj * 1024u;
#define ITER_HEAD \
            const unsigned a1 = cA + (unsigned)(t + 1) * kstep; \
            const unsigned a2 = last ? nA : cA + (unsigned)(t + 2) * kstep, b2 = last ? nB : cB + (unsigned)(t + 2) * kstep; \
            const unsigned a3 = a2 + kstep, b3 = b2 + kstep; \
            const bool fnext = t >= NKT - 4; \
            const int fi = fnext ? ni : ci, fj = fnext ? nj : cj, fq4 = ((t + 4) >> 3) & 3; \
            if ((t & 7) == 6 && wid == 0 && rdy != ~0ull && (!last || has_next)) { \
                TS0(tsk); ++tsk; \
                poll_quarter(p, call, fi, fj, ((t + 2) >> 3) & 3, badw); \
                TS0(tsk); ++tsk; \
            } \
            unsigned fv; \
            { \
                const unsigned fo = flane + (unsigned)((lane & 16) ? (NUX + fj) : fi) * 256u; \
                asm volatile("buffer_load_dword %0, %1, %2, %3 offen sc0 sc1" : "=v"(fv) : "v"(fo), "s"(rsF), "s"(fq4 * 64) : "memory"); \
            } \
            LDB(B0, 0, 0); SCHED; LDA(At, 0, 0); STAGE(SA(1, 1), a1 + hstep); \
            WAIT_L(8); BAR; WAIT_L(0); MMA(0, 0, At, B0); BAR; SCHED; \
            LDB(B1, 0, 1); STAGE(SB(0, 0), b2); \
            BAR; WAIT_L(0); MMA(0, 1, At, B1); BAR; \
            LDA(At, 0, 1); STAGE(SA(0, 0), a2); \
            BAR; WAIT_L(0); MMA(1, 0, At, B0); BAR; SCHED; \
            STAGE(SB(0, 1), b2 + hstep); \
            WAIT_V(6); \
            asm volatile("" : "+v"(fv)); \
            rdy = __ballot(fv >= call); \
            BAR; MMA(1, 1, At, B1); BAR; \
            LDB(B0, 1, 0); SCHED; LDA(At, 1, 0); STAGE(SA(0, 1), a2 + hstep); \
            WAIT_L(8); BAR; WAIT_L(0); MMA(0, 0, At, B0); BAR; SCHED;
        for (int t = 0; t < NKT - 2; t += 2) {
            constexpr bool last = false;
            ITER_HEAD
            LDB(B1, 1, 1); STAGE(SB(1, 0), b3);
            BAR; WAIT_L(0); MMA(0, 1, At, B1); BAR;
            LDA(At, 1, 1); STAGE(SA(1, 0), a3);
            BAR; WAIT_L(0); MMA(1, 0, At, B0); BAR; SCHED;
            STAGE(SB(1, 1), b3 + hstep);
            WAIT_V(6); BAR; MMA(1, 1, At, B1); BAR;
        }
        {
            constexpr bool last = true; constexpr int t = NKT - 2;
            ITER_HEAD
            unsigned zz = 0; asm volatile("" : "+v"(zz));
            const unsigned t2 = __builtin_amdgcn_mbcnt_hi(~0u, __builtin_amdgcn_mbcnt_lo(~0u, zz));
            const unsigned vo = ((unsigned)(wr * 64) + (t2 & 15)) * (N_ * 4u) + (unsigned)(wc * 128) + (t2 & 48u);
            const bool bad = __builtin_amdgcn_readfirstlane(*badw) != 0u;
            const float lo = bad ? __builtin_nanf("") : 0.f, sc2 = bad ? __builtin_nanf("") : scale;
#define EPI(ai, bj) do { \
            _Pragma("unroll") for (int m = 0; m < 4; ++m) _Pragma("unroll") for (int n = 0; n < 2; ++n) { \
                f32x4 y = acc[ai][bj][m][n] * sc2; \
                _Pragma("unroll") for (int e = 0; e < 4; ++e) asm("v_max_f32 %0, %1, %2" : "=v"(y[e]) : "s"(lo), "v"(y[e])); \
                asm volatile("buffer_store_dwordx4 %0, %1, %2, %3 offen offset:%4\n\ts_nop 1" :: "v"(y), "v"(vo), "s"(rsO), \
                             "s"(so + (unsigned)((ai) * 128 + m * 16) * (N_ * 4u) + (bj) * 512u), "n"(n * 64) : "memory"); \
                acc[ai][bj][m][n] = (f32x4){0.f, 0.f, 0.f, 0.f}; } } while (0)
            LDB(B1, 1, 1); STAGE(SB(1, 0), b3); EPI(0, 0);
            BAR; WAIT_L(0); MMA(0, 1, At, B1); BAR;
            LDA(At, 1, 1); STAGE(SA(1, 0), a3); EPI(0, 1);
            BAR; WAIT_L(0); MMA(1, 0, At, B0); BAR; SCHED;
            STAGE(SB(1, 1), b3 + hstep); EPI(1, 0);
            WAIT_V(30); BAR; MMA(1, 1, At, B1); BAR;
            EPI(1, 1);
        }
        TS0(tsk); ++tsk;
        if (!has_next) break;
        ci = ni; cj = nj; cA = nA; cB = nB; li = nl;
    }
    WAIT_V(0);
    if (wr == 0) BAR;
    BAR;
#undef SA
#undef SB
#undef STAGE
#undef LDFRAG
#undef LDA
#undef LDB
#undef MMA
#undef WAIT_V
#undef WAIT_L
#undef BAR
#undef SCHED
#undef ITER_HEAD
#undef EPI
}

__global__ __launch_bounds__(512, 2) void fused_kernel(const Params p) {
    extern __shared__ __attribute__((aligned(16))) unsigned char shm[];
    LAS unsigned char* lds = (LAS unsigned char*)shm;
    LAS unsigned* ctl = (LAS unsigned*)(lds + LDS_BYTES);
    if (threadIdx.x == 0) {
        const unsigned ticket = __hip_atomic_fetch_add((unsigned*)(p.ws[p.rank] + OFF_TICKET), 1u, __ATOMIC_RELAXED, __HIP_MEMORY_SCOPE_AGENT);
        ctl[0] = ticket / (unsigned)GRID + 1u; ctl[1] = 0u;
    }
    __syncthreads();
    const unsigned call = (unsigned)__builtin_amdgcn_readfirstlane(ctl[0]);
    const int bid = blockIdx.x;
    TS0(0);
    if (bid < NPUSH) {
        if (threadIdx.x < PWV * 64)
            push_phase(p, lds, call, (unsigned)(p.rank + 1 + (bid & 7)), (bid >> 3) * PWV + (int)__builtin_amdgcn_readfirstlane(threadIdx.x >> 6), (NPUSH / 8) * PWV);
        return;
    }
    gemm_phase(p, lds, ctl + 1, call, bid & 7, (bid - NPUSH) >> 3, (GRID - NPUSH) / 8);
}

}

extern "C" void kernel_launch(int rank, int nranks, ncclComm_t comm,
                              hipStream_t stream,
                              void* const* d_in, const int* in_sizes, int n_in,
                              void* d_out, int out_size,
                              const int* in_dtypes, int out_dtype,
                              void* const* d_out_all,
                              void* const* d_ws_all, size_t ws_bytes) {
    using namespace fk;
    if (nranks != NR || ws_bytes < WS_NEED || (unsigned)rank >= (unsigned)NR) { fprintf(stderr, "unexpected configuration\n"); return; }
    static bool attr_done = false;
    if (!attr_done) {
        HIP_CHECK(hipFuncSetAttribute((const void*)fused_kernel, hipFuncAttributeMaxDynamicSharedMemorySize, LDS_BYTES + 64));
        attr_done = true;
    }
    Params p;
    p.x = (const unsigned char*)d_in[0]; p.w = (const unsigned char*)d_in[1];
    p.sx = (const float*)d_in[2]; p.sw = (const float*)d_in[3];
    p.out = (float*)d_out; p.rank = rank; p.mode = 0;
    for (int r = 0; r < NR; ++r) p.ws[r] = (char*)d_ws_all[r];
    hipLaunchKernelGGL(fused_kernel, dim3(GRID), dim3(512), LDS_BYTES + 64, stream, p);
    HIP_CHECK(hipGetLastError());
}
```

```cpp
#include <hip/hip_runtime.h>
#include <rccl/rccl.h>
#include <cstdio>
#include <cstdint>
#include <algorithm>

#define HIP_CHECK(expr) do { hipError_t _e = (expr); if (_e != hipSuccess) { \
    fprintf(stderr, "HIP error %s at %s:%d\n", hipGetErrorString(_e), __FILE__, __LINE__); \
    return; } } while (0)

namespace fk {

#define LAS __attribute__((address_space(3)))
typedef int v8i __attribute__((ext_vector_type(8)));
typedef int v4i __attribute__((ext_vector_type(4)));
typedef unsigned v4u __attribute__((ext_vector_type(4)));
typedef float f32x4 __attribute__((ext_vector_type(4)));

constexpr int M_ = 4096, N_ = 16384, K_ = 4096, NR = 8, KSH = K_ / NR;
constexpr int NKT = K_ / 128;
constexpr unsigned BLK = 16384;
constexpr size_t XI_BYTES = (size_t)(M_ / 128) * NKT * BLK;
constexpr size_t WI_BYTES = (size_t)(N_ / 128) * NKT * BLK;
constexpr size_t BUF_BYTES = XI_BYTES + WI_BYTES;
constexpr size_t CTRL = 1u << 20;
constexpr size_t WS_NEED = CTRL + 2 * BUF_BYTES;
constexpr int NUX = M_ / 256, NUW = N_ / 256, NU = NUX + NUW;
constexpr size_t OFF_TICKET = 0, OFF_TMO = 256, OFF_BUILT = 512, OFF_FLAGS = 4096;
constexpr int GRID = 256, NITEM = NU * 8;
#define STAUX 17
#define L128 1
#define OFL "sc0 sc1 nt"
#define OST "buffer_store_dwordx4"
#define PWV 2
#define COPY 1
#define PPX 4
constexpr int LDS_BYTES = 8 * (int)BLK;
constexpr unsigned long long DEADLINE = 50000000ull;

struct Params {
    const unsigned char* x;
    const unsigned char* w;
    const float* sx; const float* sw;
    float* out;
    char* ws[NR];
    int rank;
    int mode;
};

#define TS(k) do { } while (0)
#define TS0(k) do { } while (0)
#define PM1
#define QTS
#define QM 6

__device__ __forceinline__ void item_decode(int it, int& ub, int& ks) {
    if (it < 198) {
        ks = it < 39 ? 0 : it < 85 ? 1 : it < 138 ? 2 : 3;
        ub = it - (it < 39 ? 0 : it < 85 ? 39 : it < 138 ? 85 : 138);
    } else if (it < 590) {
        const int e4 = (it - 198) / 28;
        ks = e4 & 3; ub = 32 + 7 * (e4 + 1) + (it - 198 - e4 * 28);
    } else if (it < 632) {
        ks = it < 611 ? 2 : it < 625 ? 3 : 0;
        ub = 32 + (it < 611 ? it - 485 : it < 625 ? it - 499 : it - 506);
    } else {
        const int r = it - 632;
        ub = 32 + 126 + (r & 1); ks = ((r >> 1) + (r & 1)) & 3;
    }
}

__device__ __forceinline__ char* uni(const char* q) {
    const unsigned long long a = (unsigned long long)q;
    return (char*)(((unsigned long long)(unsigned)__builtin_amdgcn_readfirstlane((unsigned)(a >> 32)) << 32) | (unsigned)__builtin_amdgcn_readfirstlane((unsigned)a));
}
#define FENCE do { asm volatile("" ::: "memory"); __builtin_amdgcn_sched_barrier(0); } while (0)

__device__ __forceinline__ void push_load(const Params& p, int it, int lane, v4u (&r)[16]) {
    int ub, ks; item_decode(it, ub, ks);
    const bool isx = ub < 32;
    const __amdgpu_buffer_rsrc_t rs = __builtin_amdgcn_make_buffer_rsrc(uni(isx ? (const char*)p.x : (const char*)p.w), 0, isx ? M_ * KSH : KSH * N_, 0x00020000);
    const unsigned so = isx ? (unsigned)(ub * 128 * KSH + ks * 128) : (unsigned)(ks * 128 * N_ + (ub - 32) * 128);
    const unsigned s2 = isx ? 16u * KSH : 2u * N_, s1 = isx ? 16u : (unsigned)N_;
    const unsigned vo = isx ? (unsigned)(lane & 15) * KSH + (unsigned)(lane >> 4) * 32u : (unsigned)(lane >> 3) * 16u * N_ + (unsigned)(lane & 7) * 16u;
#pragma unroll
    for (int q = 0; q < 16; ++q)
        r[q] = __builtin_amdgcn_raw_buffer_load_b128(rs, vo, so + (unsigned)(q >> 1) * s2 + (unsigned)(q & 1) * s1, 0);
}

__device__ __forceinline__ void push_phase(const Params& p, LAS unsigned char* lds, unsigned call, unsigned d, int pw, int npw) {
    const int lane = threadIdx.x & 63, wid = __builtin_amdgcn_readfirstlane(threadIdx.x >> 6);
    LAS unsigned char* myl = lds + wid * BLK;
    char* dbase = uni(p.ws[d & 7u]);
    const __amdgpu_buffer_rsrc_t rsD = __builtin_amdgcn_make_buffer_rsrc(dbase + CTRL + (size_t)(call & 1u) * BUF_BYTES, 0, (int)BUF_BYTES, 0x00020000);
    const __amdgpu_buffer_rsrc_t rsG = __builtin_amdgcn_make_buffer_rsrc(dbase + OFF_FLAGS, 0, NU * 256, 0x00020000);
    const unsigned lo = (unsigned)lane * 16u, fdrop = lane ? 0x40000000u : 0u;
    v4u r[16];
    push_load(p, pw, lane, r);
    asm volatile("" : "+v"(r[0]), "+v"(r[1]), "+v"(r[2]), "+v"(r[3]), "+v"(r[4]), "+v"(r[5]), "+v"(r[6]), "+v"(r[7]),
                      "+v"(r[8]), "+v"(r[9]), "+v"(r[10]), "+v"(r[11]), "+v"(r[12]), "+v"(r[13]), "+v"(r[14]), "+v"(r[15]));
    unsigned pfo = 0x40000000u;
    for (int it = pw; it < NITEM; it += npw) {
        int ub, ks; item_decode(it, ub, ks);
        v4u v[16];
        if (ub < 32) {
#pragma unroll
            for (int q = 0; q < 16; ++q) v[q] = r[q];
        } else {
            const int a = lane & 7, b = lane >> 3;
            LAS unsigned char* wl = myl + a * 2048 + (b & 1) * 1024 + (b >> 1) * 256;
#pragma unroll
            for (int wv = 0; wv < 4; ++wv) {
                v4u o[4];
#pragma unroll
                for (int g = 0; g < 4; ++g) {
                    const unsigned r0 = r[4 * g + 0][wv], r1 = r[4 * g + 1][wv], r2 = r[4 * g + 2][wv], r3 = r[4 * g + 3][wv];
                    const unsigned t0 = __builtin_amdgcn_perm(r1, r0, 0x05010400u), t1 = __builtin_amdgcn_perm(r1, r0, 0x07030602u);
                    const unsigned t2 = __builtin_amdgcn_perm(r3, r2, 0x05010400u), t3 = __builtin_amdgcn_perm(r3, r2, 0x07030602u);
                    o[0][g] = __builtin_amdgcn_perm(t2, t0, 0x05040100u);
                    o[1][g] = __builtin_amdgcn_perm(t2, t0, 0x07060302u);
                    o[2][g] = __builtin_amdgcn_perm(t3, t1, 0x05040100u);
                    o[3][g] = __builtin_amdgcn_perm(t3, t1, 0x07060302u);
                }
#pragma unroll
                for (int c = 0; c < 4; ++c) *(LAS v4u*)(wl + (4 * wv + c) * 16) = o[c];
            }
            asm volatile("s_waitcnt lgkmcnt(0)" ::: "memory");
            __builtin_amdgcn_wave_barrier();
#pragma unroll
            for (int q = 0; q < 16; ++q) v[q] = *(LAS v4u*)(myl + q * 1024 + lane * 16);
            asm volatile("s_waitcnt lgkmcnt(0)" ::: "memory");
            __builtin_amdgcn_wave_barrier();
        }
        FENCE;
        push_load(p, it + npw < NITEM ? it + npw : it, lane, r);
        FENCE;
        const unsigned base = (unsigned)(ub * NKT + ks * 8 + p.rank) * BLK;
#pragma unroll
        for (int q = 0; q < 16; ++q) __builtin_amdgcn_raw_buffer_store_b128(v[q], rsD, lo, base + q * 1024u, STAUX);
        FENCE;
        asm volatile("s_waitcnt vmcnt(33)" ::: "memory");
        __builtin_amdgcn_raw_buffer_store_b32(call, rsG, pfo | fdrop, 0, 17);
        FENCE;
        pfo = (unsigned)((ub >> 1) * 64 + ks * 16 + (ub & 1) * 8 + p.rank) * 4u;
    }
    asm volatile("s_waitcnt vmcnt(0)" ::: "memory");
    __builtin_amdgcn_raw_buffer_store_b32(call, rsG, pfo | fdrop, 0, 17);
}

__device__ __forceinline__ void poll_flags(const Params& p, unsigned call, unsigned fo, LAS unsigned* badw) {
    char* ws = p.ws[p.rank];
    const unsigned* f = (const unsigned*)(ws + OFF_FLAGS + fo);
    unsigned* tmo = (unsigned*)(ws + OFF_TMO);
    const unsigned long long t0 = __builtin_amdgcn_s_memrealtime();
    for (;;) {
        const unsigned a = __hip_atomic_load(f, __ATOMIC_RELAXED, __HIP_MEMORY_SCOPE_SYSTEM);
        if (__all(a >= call)) break;
        const unsigned tv = (unsigned)__builtin_amdgcn_readfirstlane(__hip_atomic_load(tmo, __ATOMIC_RELAXED, __HIP_MEMORY_SCOPE_AGENT));
        if (tv == call || __builtin_amdgcn_s_memrealtime() - t0 > DEADLINE) {
            if ((threadIdx.x & 63) == 0) { __hip_atomic_store(tmo, call, __ATOMIC_RELAXED, __HIP_MEMORY_SCOPE_AGENT); *badw = 1u; }
            break;
        }
        __builtin_amdgcn_s_sleep(2);
    }
    asm volatile("s_waitcnt vmcnt(0) lgkmcnt(0)" ::: "memory");
}

template <int V> struct IC { static constexpr int v = V; };
constexpr int n_epi(int F, int T, int P) { return ((T == 7 && P > ((F + 1) & 3)) || (T == 0 && P <= F)) ? 2 : 0; }
constexpr int n_ops(int F, int T, int P) { return ((P == 1 || P == 2) ? 3 : 1) + n_epi(F, T, P) + ((T == 4 && P == 0) ? 1 : 0); }
constexpr int pF(int F, int T) { return T ? F : (F + 3) & 3; }
constexpr int pT(int T) { return (T + 7) & 7; }
constexpr int w3(int F, int T) { return n_epi(pF(F, T), pT(T), 3) + n_ops(F, T, 0) + n_ops(F, T, 1) + n_ops(F, T, 2) + n_ops(F, T, 3); }
constexpr int w2(int F, int T) { return n_epi(pF(F, T), pT(T), 0) + n_ops(pF(F, T), pT(T), 1) + n_ops(pF(F, T), pT(T), 2) + n_ops(pF(F, T), pT(T), 3)
                                        + n_ops(F, T, 0) + n_ops(F, T, 1) + n_ops(F, T, 2); }

__device__ __forceinline__ void gemm_phase(const Params& p, LAS unsigned char* lds, LAS unsigned* badw, unsigned call, int ci, int grp, int l0, int lstride, int nstrip) {
    const int tid = threadIdx.x, wid = __builtin_amdgcn_readfirstlane(tid >> 6), lane = tid & 63, rg = wid >> 1, ch = wid & 1;
    char* wsl = p.ws[p.rank];
    const __amdgpu_buffer_rsrc_t rsI = __builtin_amdgcn_make_buffer_rsrc(wsl + CTRL + (size_t)(call & 1u) * BUF_BYTES, 0, (int)BUF_BYTES, 0x00020000);
    const __amdgpu_buffer_rsrc_t rsF = __builtin_amdgcn_make_buffer_rsrc(wsl + OFF_FLAGS, 0, NU * 256, 0x00020000);
    const __amdgpu_buffer_rsrc_t rsO = __builtin_amdgcn_make_buffer_rsrc(p.out, 0, (int)((size_t)M_ * N_ * 4), 0x00020000);
    const float scale = __builtin_bit_cast(float, __builtin_amdgcn_readfirstlane(__builtin_bit_cast(int, p.sx[0] * p.sw[0])));
    constexpr unsigned kstep = BLK, hstep = NKT * BLK;
    const unsigned voff = (unsigned)tid * 16u;
    const unsigned ldsw = (unsigned)wid * 1024u;
    unsigned abase = (unsigned)(size_t)lds + rg * 8192 + lane * 16, bbase = (unsigned)(size_t)lds + 32768 + ch * 4096 + lane * 16;
    unsigned abase1 = abase + 65536u, bbase1 = bbase + 65536u;
    asm volatile("" : "+v"(abase), "+v"(bbase), "+v"(abase1), "+v"(bbase1));
    unsigned vo = ((unsigned)(rg * 64) + (lane & 7)) * (N_ * 4u) + (unsigned)(ch * 128) + (lane & 48u) + (lane & 8u) * 8u;
    asm volatile("" : "+v"(vo));
#define STG(ldsoff, soff) __builtin_amdgcn_raw_ptr_buffer_load_lds(rsI, (LAS void*)(lds + (ldsoff) + ldsw), 16, voff, (soff), 0, 0)
#define STG_A(b, part, kk) do { asm volatile("" : "+s"(cA)); STG((b) * 65536 + (part) * 8192, cA + (unsigned)(((part) >> 1) * hstep + ((part) & 1) * 8192u + (kk) * kstep)); } while (0)
#define STG_B(b, f, kk) do { asm volatile("" : "+s"(bB[f])); STG((b) * 65536 + 32768 + (f) * 8192, bB[f] + (unsigned)((kk) * kstep)); } while (0)
#define LDFRAG(base, off) __builtin_shufflevector(*(const LAS v4i*)(size_t)((base) + (off)), *(const LAS v4i*)(size_t)((base) + (off) + 1024), 0, 1, 2, 3, 4, 5, 6, 7)
#define LDA(b) do { _Pragma("unroll") for (int m = 0; m < 4; ++m) At[m] = LDFRAG((b) ? abase1 : abase, m * 2048); } while (0)
#define LDB(b, f) do { _Pragma("unroll") for (int n = 0; n < 2; ++n) Bt[n] = LDFRAG((b) ? bbase1 : bbase, (f) * 8192 + n * 2048); } while (0)
#define MMA(f) do { if (__builtin_expect((act >> (f)) & 1u, 1)) { __builtin_amdgcn_s_setprio(1); _Pragma("unroll") for (int m = 0; m < 4; ++m) _Pragma("unroll") for (int n = 0; n < 2; ++n) \
        acc[f][m][n] = __builtin_amdgcn_mfma_scale_f32_16x16x128_f8f6f4(Bt[n], At[m], acc[f][m][n], 1, 0, 0, 0, 0, 0); __builtin_amdgcn_s_setprio(0); } } while (0)
#define WAIT_V(n) asm volatile("s_waitcnt vmcnt(%0)" :: "n"(n) : "memory")
#define WAIT_L(n) asm volatile("s_waitcnt lgkmcnt(" #n ")" ::: "memory")
#define BAR __builtin_amdgcn_s_barrier()
#define SCHED __builtin_amdgcn_sched_barrier(0)
    f32x4 acc[4][4][2];
#pragma unroll
    for (int a = 0; a < 4; ++a)
#pragma unroll
        for (int m = 0; m < 4; ++m)
#pragma unroll
            for (int n = 0; n < 2; ++n) acc[a][m][n] = (f32x4){0.f, 0.f, 0.f, 0.f};
    v8i At[4], Bt[2];
    unsigned cA = (unsigned)ci * 2u * hstep;
    const unsigned sB0 = (unsigned)(32 + l0) * hstep + (unsigned)grp * 8192u, sBs = (unsigned)lstride * hstep;
    unsigned bB[4] = {sB0, cA, cA, cA};
    const unsigned fx = (unsigned)ci * 256u;
    auto fw = [&](int n) { const int l = l0 + n * lstride; return (unsigned)(NUX * 256 + (l >> 1) * 256 + (l & 1) * 32); };
    unsigned fo = ((lane >> 3) == 2 ? fw(0) + (unsigned)(lane & 7) * 4u : fx + (unsigned)(lane & 15) * 4u) + (wid ? 0x40000000u : 0u);
    const unsigned sO = (unsigned)ci * (256u * N_ * 4u) + (unsigned)grp * 256u;
    unsigned eo = sO, edrop = 0x80000000u, act = 0;
    float lo = 0.f, sc2 = scale;
    unsigned long long rdy = 0;
    unsigned fv = 0;
    int tsk = 1;
    if (wid == 0) poll_flags(p, call, fo, badw);
    asm volatile("" ::: "memory"); BAR; asm volatile("" ::: "memory");
    TS0(tsk); ++tsk;
    STG_B(0, 0, 0); STG_A(0, 0, 0); STG_A(0, 1, 0); STG_A(0, 2, 0); STG_A(0, 3, 0); STG_B(0, 1, 0); STG_B(0, 2, 0); STG_B(0, 3, 0);
    STG_B(1, 0, 1); STG_A(1, 0, 1); STG_A(1, 1, 1); STG_A(1, 2, 1); STG_A(1, 3, 1); STG_B(1, 1, 1); STG_B(1, 2, 1);
    WAIT_V(0); BAR;
    if (wid >= 4) BAR;
    int e = 0;
#define EPI(f, m) do { f32x4 y[2]; const unsigned vq = vo | edrop; \
        _Pragma("unroll") for (int c = 0; c < 4; ++c) { \
            const float f0 = acc[f][m][0][c], f1 = acc[f][m][1][c]; \
            const int x0 = __builtin_bit_cast(int, f0), x1 = __builtin_bit_cast(int, f1); \
            y[0][c] = __builtin_bit_cast(float, __builtin_amdgcn_update_dpp(x0, x1, 0x128, 0xf, 0xc, false)) * sc2; \
            y[1][c] = __builtin_bit_cast(float, __builtin_amdgcn_update_dpp(x1, x0, 0x128, 0xf, 0x3, false)) * sc2; } \
        _Pragma("unroll") for (int n = 0; n < 2; ++n) { \
            _Pragma("unroll") for (int c = 0; c < 4; ++c) asm("v_max_f32 %0, %1, %2" : "=v"(y[n][c]) : "s"(lo), "v"(y[n][c])); \
            asm volatile(OST " %0, %1, %2, %3 offen " OFL "\n\ts_nop 1" :: "v"(y[n]), "v"(vq), "s"(rsO), "s"(eo + (unsigned)((m) * 16 + n * 8) * (N_ * 4u)) : "memory"); } \
        acc[f][m][0] = (f32x4){0.f, 0.f, 0.f, 0.f}; acc[f][m][1] = (f32x4){0.f, 0.f, 0.f, 0.f}; } while (0)
#define EPI_AT(P) do { if constexpr (T == 7 && (P) > G) { EPI(G, ((P) - G - 1) & 3); } else if constexpr (T == 0 && (P) <= F) { EPI(F, (3 - F + (P)) & 3); } } while (0)
    auto step = [&](auto Fc, auto Tc) __attribute__((always_inline)) {
        constexpr int F = decltype(Fc)::v, T = decltype(Tc)::v, b = T & 1, G = (F + 1) & 3;
        constexpr int k1 = (F * 8 + T + 1) & 31, k2 = (F * 8 + T + 2) & 31;
        if constexpr (T == 0) { act = (act & ~(1u << F)) | (e < nstrip ? 1u << F : 0u); TS0(tsk); ++tsk; }
        if constexpr (T == 4) {
            const unsigned fn = e + 1 < nstrip ? fw(e + 1) : fx;
            if ((lane >> 3) == 2 + G) fo = fn + (unsigned)(lane & 7) * 4u + (wid ? 0x40000000u : 0u);
            asm volatile("buffer_load_dword %0, %1, %2, 0 offen offset:%3 sc0 sc1" : "=v"(fv) : "v"(fo), "s"(rsF), "n"(G * 64) : "memory");
        }
        if constexpr (T == 6) {
            if (wid == 0 && rdy != ~0ull) { poll_flags(p, call, fo + G * 64u, badw); }
        }
        if constexpr (T == 7) {
            const bool bad = __builtin_amdgcn_readfirstlane(*badw) != 0u;
            lo = bad ? __builtin_nanf("") : 0.f; sc2 = bad ? __builtin_nanf("") : scale;
        }
        LDB(b, 0); SCHED; LDA(b); STG_B(b ^ 1, 3, k1); EPI_AT(0);
        WAIT_L(8); BAR; WAIT_L(0); MMA(0); BAR; SCHED;
        if constexpr (T == 6) {
            edrop = (e >= 3 && e - 3 < nstrip) ? 0u : 0x80000000u;
            eo = sO + (unsigned)(l0 + (e >= 3 ? e - 3 : 0) * lstride) * 512u;
            bB[G] = e + 1 < nstrip ? sB0 + (unsigned)(e + 1) * sBs : cA;
        }
        LDB(b, 1); STG_B(b, 0, k2); STG_A(b, 0, k2); STG_A(b, 1, k2); EPI_AT(1);
        WAIT_L(0); BAR; MMA(1); BAR; SCHED;
        LDB(b, 2); STG_B(b, 1, k2); STG_A(b, 2, k2); STG_A(b, 3, k2); EPI_AT(2);
        WAIT_V(w2(F, T)); WAIT_L(0); BAR; MMA(2); BAR; SCHED;
        LDB(b, 3); STG_B(b, 2, k2); EPI_AT(3);
        WAIT_V(w3(F, T));
        if constexpr (T == 5) { asm volatile("" : "+v"(fv)); rdy = __ballot(fv >= call); PM1 }
        WAIT_L(0); BAR; MMA(3); BAR; SCHED;
    };
#define EPOCH(F) \
        step(IC<F>{}, IC<0>{}); \
        if (e == nstrip + 3) break; \
        step(IC<F>{}, IC<1>{}); step(IC<F>{}, IC<2>{}); step(IC<F>{}, IC<3>{}); step(IC<F>{}, IC<4>{}); \
        step(IC<F>{}, IC<5>{}); step(IC<F>{}, IC<6>{}); step(IC<F>{}, IC<7>{}); ++e;
    for (;;) { EPOCH(0) EPOCH(1) EPOCH(2) EPOCH(3) }
    TS0(tsk); ++tsk;
    WAIT_V(0);
    if (wid < 4) BAR;
#undef STG
#undef STG_A
#undef STG_B
#undef LDFRAG
#undef LDA
#undef LDB
#undef MMA
#undef WAIT_V
#undef WAIT_L
#undef BAR
#undef SCHED
#undef EPI
#undef EPI_AT
#undef EPOCH
}

__global__ __launch_bounds__(512, 2) void fused_kernel(const Params p) {
    extern __shared__ __attribute__((aligned(16))) unsigned char shm[];
    LAS unsigned char* lds = (LAS unsigned char*)shm;
    LAS unsigned* ctl = (LAS unsigned*)(lds + LDS_BYTES);
    if (threadIdx.x == 0) {
        const unsigned ticket = __hip_atomic_fetch_add((unsigned*)(p.ws[p.rank] + OFF_TICKET), 1u, __ATOMIC_RELAXED, __HIP_MEMORY_SCOPE_AGENT);
        ctl[0] = ticket / (unsigned)GRID + 1u; ctl[1] = 0u;
    }
    __syncthreads();
    const unsigned call = (unsigned)__builtin_amdgcn_readfirstlane(ctl[0]);
    const int bid = blockIdx.x;
    TS0(0);
    const int xcd = bid & 7, slot = bid >> 3, wid = __builtin_amdgcn_readfirstlane(threadIdx.x >> 6);
    const bool pusher = slot < 4;
    if (pusher) {
        if (wid < PWV) push_phase(p, lds, call, (unsigned)(p.rank + xcd), slot * PWV + wid, 4 * PWV);
        asm volatile("s_waitcnt vmcnt(0)" ::: "memory");
        __syncthreads();
    }
    TS0(31);
    gemm_phase(p, lds, ctl + 1, call, (xcd & 3) * 4 + (slot & 3), xcd >> 2, pusher ? 126 : (slot >> 2) - 1, pusher ? 1 : 7, pusher ? 2 : 18);
}

}

extern "C" void kernel_launch(int rank, int nranks, ncclComm_t comm,
                              hipStream_t stream,
                              void* const* d_in, const int* in_sizes, int n_in,
                              void* d_out, int out_size,
                              const int* in_dtypes, int out_dtype,
                              void* const* d_out_all,
                              void* const* d_ws_all, size_t ws_bytes) {
    using namespace fk;
    if (nranks != NR || ws_bytes < WS_NEED || (unsigned)rank >= (unsigned)NR) { fprintf(stderr, "unexpected configuration\n"); return; }
    static bool attr_done = false;
    if (!attr_done) {
        HIP_CHECK(hipFuncSetAttribute((const void*)fused_kernel, hipFuncAttributeMaxDynamicSharedMemorySize, LDS_BYTES + 64));
        attr_done = true;
    }
    Params p;
    p.x = (const unsigned char*)d_in[0]; p.w = (const unsigned char*)d_in[1];
    p.sx = (const float*)d_in[2]; p.sw = (const float*)d_in[3];
    p.out = (float*)d_out; p.rank = rank; p.mode = 0;
    for (int r = 0; r < NR; ++r) p.ws[r] = (char*)d_ws_all[r];
    hipLaunchKernelGGL(fused_kernel, dim3(GRID), dim3(512), LDS_BYTES + 64, stream, p);
    HIP_CHECK(hipGetLastError());
}
```

```cpp
#include <hip/hip_runtime.h>
#include <rccl/rccl.h>
#include <cstdio>
#include <cstdint>
#include <algorithm>

#define HIP_CHECK(expr) do { hipError_t _e = (expr); if (_e != hipSuccess) { \
    fprintf(stderr, "HIP error %s at %s:%d\n", hipGetErrorString(_e), __FILE__, __LINE__); \
    return; } } while (0)

namespace fk {

#define LAS __attribute__((address_space(3)))
typedef int v8i __attribute__((ext_vector_type(8)));
typedef int v4i __attribute__((ext_vector_type(4)));
typedef unsigned v4u __attribute__((ext_vector_type(4)));
typedef float f32x4 __attribute__((ext_vector_type(4)));
typedef float f32x2 __attribute__((ext_vector_type(2)));

constexpr int M_ = 4096, N_ = 16384, K_ = 4096, NR = 8, KSH = K_ / NR;
constexpr int NKT = K_ / 128;
constexpr unsigned BLK = 16384;
constexpr size_t XI_BYTES = (size_t)(M_ / 128) * NKT * BLK;
constexpr size_t WI_BYTES = (size_t)(N_ / 128) * NKT * BLK;
constexpr size_t BUF_BYTES = XI_BYTES + WI_BYTES;
constexpr size_t CTRL = 1u << 20;
constexpr size_t WS_NEED = CTRL + 2 * BUF_BYTES;
constexpr int NUX = M_ / 256, NUW = N_ / 256, NU = NUX + NUW;
constexpr size_t OFF_TICKET = 0, OFF_TMO = 256, OFF_BUILT = 512, OFF_FLAGS = 4096;
constexpr int GRID = 256, NITEM = NU * 8;
#define STAUX 17
#define L128 1
#define OFL "nt"
#define OST "buffer_store_dwordx4"
#define PWV 2
#define COPY 1
#define PPX 4
constexpr int LDS_BYTES = 8 * (int)BLK;
constexpr unsigned long long DEADLINE = 50000000ull;

struct Params {
    const unsigned char* x;
    const unsigned char* w;
    const float* sx; const float* sw;
    float* out;
    char* ws[NR];
    int rank;
    int mode;
};

#define TS(k) do { } while (0)
#define TS0(k) do { } while (0)
#define PM1
#define QTS
#define QM 6

__device__ __forceinline__ void item_decode(int it, int& ub, int& ks) {
    if (it < 198) {
        ks = it < 39 ? 0 : it < 85 ? 1 : it < 138 ? 2 : 3;
        ub = it - (it < 39 ? 0 : it < 85 ? 39 : it < 138 ? 85 : 138);
    } else if (it < 590) {
        const int e4 = (it - 198) / 28;
        ks = e4 & 3; ub = 32 + 7 * (e4 + 1) + (it - 198 - e4 * 28);
    } else if (it < 632) {
        ks = it < 611 ? 2 : it < 625 ? 3 : 0;
        ub = 32 + (it < 611 ? it - 485 : it < 625 ? it - 499 : it - 506);
    } else {
        const int r = it - 632;
        ub = 32 + 126 + (r & 1); ks = ((r >> 1) + (r & 1)) & 3;
    }
}

__device__ __forceinline__ char* uni(const char* q) {
    const unsigned long long a = (unsigned long long)q;
    return (char*)(((unsigned long long)(unsigned)__builtin_amdgcn_readfirstlane((unsigned)(a >> 32)) << 32) | (unsigned)__builtin_amdgcn_readfirstlane((unsigned)a));
}
#define FENCE do { asm volatile("" ::: "memory"); __builtin_amdgcn_sched_barrier(0); } while (0)

__device__ __forceinline__ void push_load(const Params& p, int it, int lane, v4u (&r)[16]) {
    int ub, ks; item_decode(it, ub, ks);
    const bool isx = ub < 32;
    const __amdgpu_buffer_rsrc_t rs = __builtin_amdgcn_make_buffer_rsrc(uni(isx ? (const char*)p.x : (const char*)p.w), 0, isx ? M_ * KSH : KSH * N_, 0x00020000);
    const unsigned so = isx ? (unsigned)(ub * 128 * KSH + ks * 128) : (unsigned)(ks * 128 * N_ + (ub - 32) * 128);
    const unsigned s2 = isx ? 16u * KSH : 2u * N_, s1 = isx ? 16u : (unsigned)N_;
    const unsigned vo = isx ? (unsigned)(lane & 15) * KSH + (unsigned)(lane >> 4) * 32u : (unsigned)(lane >> 3) * 16u * N_ + (unsigned)(lane & 7) * 16u;
#pragma unroll
    for (int q = 0; q < 16; ++q)
        r[q] = __builtin_amdgcn_raw_buffer_load_b128(rs, vo, so + (unsigned)(q >> 1) * s2 + (unsigned)(q & 1) * s1, 0);
}

__device__ __forceinline__ void push_phase(const Params& p, LAS unsigned char* lds, unsigned call, unsigned d, int pw, int npw) {
    const int lane = threadIdx.x & 63, wid = __builtin_amdgcn_readfirstlane(threadIdx.x >> 6);
    LAS unsigned char* myl = lds + wid * BLK;
    char* dbase = uni(p.ws[d & 7u]);
    const __amdgpu_buffer_rsrc_t rsD = __builtin_amdgcn_make_buffer_rsrc(dbase + CTRL + (size_t)(call & 1u) * BUF_BYTES, 0, (int)BUF_BYTES, 0x00020000);
    const __amdgpu_buffer_rsrc_t rsG = __builtin_amdgcn_make_buffer_rsrc(dbase + OFF_FLAGS, 0, NU * 256, 0x00020000);
    const unsigned lo = (unsigned)lane * 16u, fdrop = lane ? 0x40000000u : 0u;
    v4u r[16];
    push_load(p, pw, lane, r);
    asm volatile("" : "+v"(r[0]), "+v"(r[1]), "+v"(r[2]), "+v"(r[3]), "+v"(r[4]), "+v"(r[5]), "+v"(r[6]), "+v"(r[7]),
                      "+v"(r[8]), "+v"(r[9]), "+v"(r[10]), "+v"(r[11]), "+v"(r[12]), "+v"(r[13]), "+v"(r[14]), "+v"(r[15]));
    unsigned pfo = 0x40000000u;
    for (int it = pw; it < NITEM; it += npw) {
        int ub, ks; item_decode(it, ub, ks);
        v4u v[16];
        if (ub < 32) {
#pragma unroll
            for (int q = 0; q < 16; ++q) v[q] = r[q];
        } else {
            const int a = lane & 7, b = lane >> 3;
            LAS unsigned char* wl = myl + a * 2048 + (b & 1) * 1024 + (b >> 1) * 256;
#pragma unroll
            for (int wv = 0; wv < 4; ++wv) {
                v4u o[4];
#pragma unroll
                for (int g = 0; g < 4; ++g) {
                    const unsigned r0 = r[4 * g + 0][wv], r1 = r[4 * g + 1][wv], r2 = r[4 * g + 2][wv], r3 = r[4 * g + 3][wv];
                    const unsigned t0 = __builtin_amdgcn_perm(r1, r0, 0x05010400u), t1 = __builtin_amdgcn_perm(r1, r0, 0x07030602u);
                    const unsigned t2 = __builtin_amdgcn_perm(r3, r2, 0x05010400u), t3 = __builtin_amdgcn_perm(r3, r2, 0x07030602u);
                    o[0][g] = __builtin_amdgcn_perm(t2, t0, 0x05040100u);
                    o[1][g] = __builtin_amdgcn_perm(t2, t0, 0x07060302u);
                    o[2][g] = __builtin_amdgcn_perm(t3, t1, 0x05040100u);
                    o[3][g] = __builtin_amdgcn_perm(t3, t1, 0x07060302u);
                }
#pragma unroll
                for (int c = 0; c < 4; ++c) *(LAS v4u*)(wl + (4 * wv + c) * 16) = o[c];
            }
            asm volatile("s_waitcnt lgkmcnt(0)" ::: "memory");
            __builtin_amdgcn_wave_barrier();
#pragma unroll
            for (int q = 0; q < 16; ++q) v[q] = *(LAS v4u*)(myl + q * 1024 + lane * 16);
            asm volatile("s_waitcnt lgkmcnt(0)" ::: "memory");
            __builtin_amdgcn_wave_barrier();
        }
        FENCE;
        push_load(p, it + npw < NITEM ? it + npw : it, lane, r);
        FENCE;
        const unsigned base = (unsigned)(ub * NKT + ks * 8 + p.rank) * BLK;
#pragma unroll
        for (int q = 0; q < 16; ++q) __builtin_amdgcn_raw_buffer_store_b128(v[q], rsD, lo, base + q * 1024u, STAUX);
        FENCE;
        asm volatile("s_waitcnt vmcnt(33)" ::: "memory");
        __builtin_amdgcn_raw_buffer_store_b32(call, rsG, pfo | fdrop, 0, 17);
        FENCE;
        pfo = (unsigned)((ub >> 1) * 64 + ks * 16 + (ub & 1) * 8 + p.rank) * 4u;
    }
    asm volatile("s_waitcnt vmcnt(0)" ::: "memory");
    __builtin_amdgcn_raw_buffer_store_b32(call, rsG, pfo | fdrop, 0, 17);
}

__device__ __forceinline__ void poll_flags(const Params& p, unsigned call, unsigned fo, LAS unsigned* badw) {
    char* ws = p.ws[p.rank];
    const unsigned* f = (const unsigned*)(ws + OFF_FLAGS + fo);
    unsigned* tmo = (unsigned*)(ws + OFF_TMO);
    const unsigned long long t0 = __builtin_amdgcn_s_memrealtime();
    for (;;) {
        const unsigned a = __hip_atomic_load(f, __ATOMIC_RELAXED, __HIP_MEMORY_SCOPE_SYSTEM);
        if (__all(a >= call)) break;
        const unsigned tv = (unsigned)__builtin_amdgcn_readfirstlane(__hip_atomic_load(tmo, __ATOMIC_RELAXED, __HIP_MEMORY_SCOPE_AGENT));
        if (tv == call || __builtin_amdgcn_s_memrealtime() - t0 > DEADLINE) {
            if ((threadIdx.x & 63) == 0) { __hip_atomic_store(tmo, call, __ATOMIC_RELAXED, __HIP_MEMORY_SCOPE_AGENT); *badw = 1u; }
            break;
        }
        __builtin_amdgcn_s_sleep(2);
    }
    asm volatile("s_waitcnt vmcnt(0) lgkmcnt(0)" ::: "memory");
}

template <int V> struct IC { static constexpr int v = V; };
constexpr int n_ops(int T, int P) { return P == 0 ? (((T & 7) == 4) ? 2 : 1) : P == 3 ? 2 : 3; }
constexpr int n_all(int T) { return n_ops(T, 0) + n_ops(T, 1) + n_ops(T, 2) + n_ops(T, 3); }
constexpr int w1(int T) { return 1 + n_all(T + 7) + n_ops(T, 0) + n_ops(T, 1); }
constexpr int w2(int T) { return n_all(T + 7) - n_ops(T + 7, 0) + n_all(T) - n_ops(T, 3); }
constexpr int w3(int T) { return n_ops(T + 7, 3) + n_all(T); }

__device__ __forceinline__ void gemm_phase(const Params& p, LAS unsigned char* lds, LAS unsigned* badw, unsigned call, int ci, int grp, int l0, int lstride, int nstrip) {
    const int tid = threadIdx.x, wid = __builtin_amdgcn_readfirstlane(tid >> 6), lane = tid & 63, rg = wid >> 1, ch = wid & 1;
    char* wsl = p.ws[p.rank];
    const __amdgpu_buffer_rsrc_t rsI = __builtin_amdgcn_make_buffer_rsrc(wsl + CTRL + (size_t)(call & 1u) * BUF_BYTES, 0, (int)BUF_BYTES, 0x00020000);
    const __amdgpu_buffer_rsrc_t rsF = __builtin_amdgcn_make_buffer_rsrc(wsl + OFF_FLAGS, 0, NU * 256, 0x00020000);
    const __amdgpu_buffer_rsrc_t rsO = __builtin_amdgcn_make_buffer_rsrc(p.out, 0, (int)((size_t)M_ * N_ * 4), 0x00020000);
    const unsigned scale = (unsigned)__builtin_amdgcn_readfirstlane(__builtin_bit_cast(int, p.sx[0] * p.sw[0]));
    constexpr unsigned kstep = BLK, hstep = NKT * BLK;
    const unsigned voff = (unsigned)tid * 16u;
    const unsigned ldsw = (unsigned)wid * 1024u;
    unsigned abase = (unsigned)(size_t)lds + rg * 8192 + lane * 16, bbase = (unsigned)(size_t)lds + 65536 + ch * 4096 + lane * 16;
    asm volatile("" : "+v"(abase), "+v"(bbase));
    unsigned vq = ((unsigned)(rg * 64) + (lane & 7)) * (N_ * 4u) + (unsigned)(ch * 128) + (lane & 48u) + (lane & 8u) * 8u + 0x80000000u;
    asm volatile("" : "+v"(vq));
#define STG(ldsoff, soff) __builtin_amdgcn_raw_ptr_buffer_load_lds(rsI, (LAS void*)(lds + (ldsoff) + ldsw), 16, voff, (soff), 0, 0)
#define STG_A(b, part, kk) do { asm volatile("" : "+s"(cA)); STG((b) * 32768 + (part) * 8192, cA + (unsigned)(((part) >> 1) * hstep + ((part) & 1) * 8192u + (kk) * kstep)); } while (0)
#define STG_B(b, f, kk) do { asm volatile("" : "+s"(bB[f])); STG(65536 + (b) * 32768 + (f) * 8192, bB[f] + (unsigned)((kk) * kstep)); } while (0)
#define LDFRAG(base, off) __builtin_shufflevector(*(const LAS v4i*)(size_t)((base) + (off)), *(const LAS v4i*)(size_t)((base) + (off) + 1024), 0, 1, 2, 3, 4, 5, 6, 7)
#define LDA(b) do { _Pragma("unroll") for (int m = 0; m < 4; ++m) At[m] = LDFRAG(abase, (b) * 32768 + m * 2048); } while (0)
#define LDB(b, f) do { _Pragma("unroll") for (int n = 0; n < 2; ++n) Bt[n] = LDFRAG(bbase, (b) * 32768 + (f) * 8192 + n * 2048); } while (0)
#define WAIT_V(n) asm volatile("s_waitcnt vmcnt(%0)" :: "n"(n) : "memory")
#define WAIT_L(n) asm volatile("s_waitcnt lgkmcnt(" #n ")" ::: "memory")
#define BAR __builtin_amdgcn_s_barrier()
#define SCHED __builtin_amdgcn_sched_barrier(0)
    f32x4 acc[4][4][2];
    f32x4 yy[4][2];
#pragma unroll
    for (int m = 0; m < 4; ++m)
#pragma unroll
        for (int n = 0; n < 2; ++n) {
            yy[m][n] = (f32x4){0.f, 0.f, 0.f, 0.f};
#pragma unroll
            for (int a = 0; a < 4; ++a) acc[a][m][n] = (f32x4){0.f, 0.f, 0.f, 0.f};
        }
    v8i At[4], Bt[2];
    unsigned cA = (unsigned)ci * 2u * hstep;
    const unsigned sB0 = (unsigned)(32 + l0) * hstep + (unsigned)grp * 8192u, sBs = (unsigned)lstride * hstep;
    unsigned bB[4] = {sB0, cA, cA, cA};
    const unsigned fx = (unsigned)ci * 256u;
    auto fw = [&](int n) { const int l = l0 + n * lstride; return (unsigned)(NUX * 256 + (l >> 1) * 256 + (l & 1) * 32); };
    unsigned fo = ((lane >> 3) == 2 ? fw(0) + (unsigned)(lane & 7) * 4u : fx + (unsigned)(lane & 15) * 4u) + (wid ? 0x40000000u : 0u);
    const unsigned sO = (unsigned)ci * (256u * N_ * 4u) + (unsigned)grp * 256u;
    unsigned eo = sO, act = 0;
    float lo = 0.f;
    unsigned long long sc2 = 0;
    unsigned long long rdy = 0;
    unsigned fv = 0;
    int tsk = 1;
    if (wid == 0) poll_flags(p, call, fo, badw);
    asm volatile("" ::: "memory"); BAR; asm volatile("" ::: "memory");
    TS0(tsk); ++tsk;
    STG_B(0, 0, 0); STG_A(0, 0, 0); STG_A(0, 1, 0); STG_A(0, 2, 0); STG_A(0, 3, 0); STG_B(0, 1, 0); STG_B(0, 2, 0); STG_B(0, 3, 0);
    STG_B(1, 0, 1); STG_A(1, 0, 1); STG_A(1, 1, 1); STG_A(1, 2, 1); STG_A(1, 3, 1); STG_B(1, 1, 1); STG_B(1, 2, 1);
    WAIT_V(0); BAR;
    if (wid >= 4) BAR;
    int e = 0;
    auto chunk = [&](auto Gc, auto Jc) __attribute__((always_inline)) {
        constexpr int G = decltype(Gc)::v, J = decltype(Jc)::v, m = (J / 5) & 3, r = J % 5;
        if constexpr (J >= 0 && J < 20) {
            if constexpr (r == 0) {
#pragma unroll
                for (int n = 0; n < 2; ++n) {
                    f32x2 h0 = __builtin_shufflevector(acc[G][m][n], acc[G][m][n], 0, 1), h1 = __builtin_shufflevector(acc[G][m][n], acc[G][m][n], 2, 3);
                    asm volatile("v_pk_mul_f32 %0, %0, %1" : "+v"(h0) : "s"(sc2));
                    asm volatile("v_pk_mul_f32 %0, %0, %1" : "+v"(h1) : "s"(sc2));
                    acc[G][m][n] = __builtin_shufflevector(h0, h1, 0, 1, 2, 3);
                }
            } else if constexpr (r <= 2) {
#pragma unroll
                for (int c = 0; c < 4; ++c) asm volatile("v_max_f32 %0, %1, %2" : "=v"(yy[m][r - 1][c]) : "v"(lo), "v"(acc[G][m][r - 1][c]));
            } else if constexpr (r == 3) {
#pragma unroll
                for (int c = 0; c < 4; ++c) asm volatile("v_max_f32_dpp %0, %1, %2 row_ror:8 row_mask:0xf bank_mask:0xc" : "+v"(yy[m][0][c]) : "v"(acc[G][m][1][c]), "v"(lo));
            } else {
#pragma unroll
                for (int c = 0; c < 4; ++c) asm volatile("v_max_f32_dpp %0, %1, %2 row_ror:8 row_mask:0xf bank_mask:0x3" : "+v"(yy[m][1][c]) : "v"(acc[G][m][0][c]), "v"(lo));
            }
        }
    };
    auto mma = [&](auto fc, auto Zc, auto Gc, auto Jc) __attribute__((always_inline)) {
        constexpr int f = decltype(fc)::v, J0 = decltype(Jc)::v;
        constexpr bool Z = decltype(Zc)::v != 0;
        if (J0 >= 0 || __builtin_expect((act >> f) & 1u, 1)) {
            __builtin_amdgcn_s_setprio(1);
#define MF(i) do { constexpr int m = (i) >> 1, n = (i) & 1; \
            if constexpr (Z) asm volatile("v_mfma_f32_16x16x128_f8f6f4 %0, %1, %2, 0 cbsz:1" : "+v"(acc[f][m][n]) : "v"(Bt[n]), "v"(At[m])); \
            else asm volatile("v_mfma_f32_16x16x128_f8f6f4 %0, %1, %2, %0 cbsz:1" : "+v"(acc[f][m][n]) : "v"(Bt[n]), "v"(At[m])); \
            if constexpr (J0 >= 0 && (i) < 7) chunk(Gc, IC<J0 + (i)>{}); } while (0)
            MF(0); MF(1); MF(2); MF(3); MF(4); MF(5); MF(6); MF(7);
#undef MF
            __builtin_amdgcn_s_setprio(0);
        }
    };
#define OUT(j) asm volatile(OST " %0, %1, %2, %3 offen " OFL "\n\ts_nop 1" :: "v"(yy[(j) >> 1][(j) & 1]), "v"(vq), "s"(rsO), "s"(eo + (unsigned)((j) * 8) * (N_ * 4u)) : "memory")
#define MMA(P) mma(IC<P>{}, IC<(T == 0 && (P) == F)>{}, IC<(T == 7 ? G : F)>{}, \
                   IC<((T == 7 && (P) > G) ? ((P) - G - 1) * 7 : (T == 0 && (P) < F) ? (3 - F + (P)) * 7 : -1)>{})
    auto step = [&](auto Fc, auto Tc) __attribute__((always_inline)) {
        constexpr int F = decltype(Fc)::v, T = decltype(Tc)::v, b = T & 1, G = (F + 1) & 3;
        constexpr int k1 = (F * 8 + T + 1) & 31, k2 = (F * 8 + T + 2) & 31;
        if constexpr (T == 0) {
            act = (act & ~(1u << F)) | (e < nstrip ? 1u << F : 0u);
            vq = (vq & 0x7fffffffu) | ((e >= 4 && e - 4 < nstrip) ? 0u : 0x80000000u);
            eo = sO + (unsigned)(l0 + (e >= 4 ? e - 4 : 0) * lstride) * 512u;
        }
        if constexpr (T == 0) { TS0(tsk); ++tsk; }
        if constexpr (T == 4) {
            const unsigned fn = e + 1 < nstrip ? fw(e + 1) : fx;
            if ((lane >> 3) == 2 + G) fo = fn + (unsigned)(lane & 7) * 4u + (wid ? 0x40000000u : 0u);
            asm volatile("buffer_load_dword %0, %1, %2, 0 offen offset:%3 sc0 sc1" : "=v"(fv) : "v"(fo), "s"(rsF), "n"(G * 64) : "memory");
        }
        if constexpr (T == 6) {
            if (wid == 0 && rdy != ~0ull) { poll_flags(p, call, fo + G * 64u, badw); }
        }
        if constexpr (T == 7) {
            const bool bad = __builtin_amdgcn_readfirstlane(*badw) != 0u;
            const unsigned s1 = bad ? 0x7fc00000u : scale;
            lo = bad ? __builtin_nanf("") : 0.f; sc2 = ((unsigned long long)s1 << 32) | s1;
            asm volatile("" : "+v"(lo), "+s"(sc2));
        }
        LDB(b, 0); SCHED; LDA(b); STG_B(b ^ 1, 3, k1);
        WAIT_L(8); BAR; WAIT_L(0); MMA(0); BAR; SCHED;
        if constexpr (T == 6) {
            bB[G] = e + 1 < nstrip ? sB0 + (unsigned)(e + 1) * sBs : cA;
        }
        LDB(b, 1); STG_B(b, 0, k2); STG_A(b, 0, k2); STG_A(b, 1, k2);
        WAIT_V(w1(T)); WAIT_L(0); BAR; MMA(1); BAR; SCHED;
        LDB(b, 2); STG_B(b, 1, k2); STG_A(b, 2, k2); STG_A(b, 3, k2);
        WAIT_V(w2(T)); WAIT_L(0); BAR; MMA(2); BAR; SCHED;
        LDB(b, 3); STG_B(b, 2, k2); OUT(T);
        WAIT_V(w3(T));
        if constexpr (T == 5) { asm volatile("" : "+v"(fv)); rdy = __ballot(fv >= call); PM1 }
        WAIT_L(0); BAR; MMA(3); BAR; SCHED;
    };
#define EPOCH(F) \
        step(IC<F>{}, IC<0>{}); \
        if (e == nstrip + 3) break; \
        step(IC<F>{}, IC<1>{}); step(IC<F>{}, IC<2>{}); step(IC<F>{}, IC<3>{}); step(IC<F>{}, IC<4>{}); \
        step(IC<F>{}, IC<5>{}); step(IC<F>{}, IC<6>{}); step(IC<F>{}, IC<7>{}); ++e;
    for (;;) { EPOCH(0) EPOCH(1) EPOCH(2) EPOCH(3) }
    OUT(1); OUT(2); OUT(3); OUT(4); OUT(5); OUT(6); OUT(7);
    TS0(tsk); ++tsk;
    WAIT_V(0);
    if (wid < 4) BAR;
#undef STG
#undef STG_A
#undef STG_B
#undef LDFRAG
#undef LDA
#undef LDB
#undef MMA
#undef WAIT_V
#undef WAIT_L
#undef BAR
#undef SCHED
#undef OUT
#undef EPOCH
}

__global__ __launch_bounds__(512, 2) void fused_kernel(const Params p) {
    extern __shared__ __attribute__((aligned(16))) unsigned char shm[];
    LAS unsigned char* lds = (LAS unsigned char*)shm;
    LAS unsigned* ctl = (LAS unsigned*)(lds + LDS_BYTES);
    if (threadIdx.x == 0) {
        const unsigned ticket = __hip_atomic_fetch_add((unsigned*)(p.ws[p.rank] + OFF_TICKET), 1u, __ATOMIC_RELAXED, __HIP_MEMORY_SCOPE_AGENT);
        ctl[0] = ticket / (unsigned)GRID + 1u; ctl[1] = 0u;
    }
    __syncthreads();
    const unsigned call = (unsigned)__builtin_amdgcn_readfirstlane(ctl[0]);
    const int bid = blockIdx.x;
    TS0(0);
    const int xcd = bid & 7, slot = bid >> 3, wid = __builtin_amdgcn_readfirstlane(threadIdx.x >> 6);
    const bool pusher = slot < 4;
    if (pusher) {
        if (wid < PWV) push_phase(p, lds, call, (unsigned)(p.rank + xcd), slot * PWV + wid, 4 * PWV);
        asm volatile("s_waitcnt vmcnt(0)" ::: "memory");
        __syncthreads();
    }
    TS0(31);
    gemm_phase(p, lds, ctl + 1, call, (xcd & 3) * 4 + (slot & 3), xcd >> 2, pusher ? 126 : (slot >> 2) - 1, pusher ? 1 : 7, pusher ? 2 : 18);
}

}

extern "C" void kernel_launch(int rank, int nranks, ncclComm_t comm,
                              hipStream_t stream,
                              void* const* d_in, const int* in_sizes, int n_in,
                              void* d_out, int out_size,
                              const int* in_dtypes, int out_dtype,
                              void* const* d_out_all,
                              void* const* d_ws_all, size_t ws_bytes) {
    using namespace fk;
    if (nranks != NR || ws_bytes < WS_NEED || (unsigned)rank >= (unsigned)NR) { fprintf(stderr, "unexpected configuration\n"); return; }
    static bool attr_done = false;
    if (!attr_done) {
        HIP_CHECK(hipFuncSetAttribute((const void*)fused_kernel, hipFuncAttributeMaxDynamicSharedMemorySize, LDS_BYTES + 64));
        attr_done = true;
    }
    Params p;
    p.x = (const unsigned char*)d_in[0]; p.w = (const unsigned char*)d_in[1];
    p.sx = (const float*)d_in[2]; p.sw = (const float*)d_in[3];
    p.out = (float*)d_out; p.rank = rank; p.mode = 0;
    for (int r = 0; r < NR; ++r) p.ws[r] = (char*)d_ws_all[r];
    hipLaunchKernelGGL(fused_kernel, dim3(GRID), dim3(512), LDS_BYTES + 64, stream, p);
    HIP_CHECK(hipGetLastError());
}
```

```cpp
#include <hip/hip_runtime.h>
#include <rccl/rccl.h>
#include <cstdio>
#include <cstdint>
#include <algorithm>

#define HIP_CHECK(expr) do { hipError_t _e = (expr); if (_e != hipSuccess) { \
    fprintf(stderr, "HIP error %s at %s:%d\n", hipGetErrorString(_e), __FILE__, __LINE__); \
    return; } } while (0)

namespace fk {

#define LAS __attribute__((address_space(3)))
typedef int v8i __attribute__((ext_vector_type(8)));
typedef int v4i __attribute__((ext_vector_type(4)));
typedef unsigned v4u __attribute__((ext_vector_type(4)));
typedef float f32x4 __attribute__((ext_vector_type(4)));
typedef float f32x2 __attribute__((ext_vector_type(2)));

constexpr int M_ = 4096, N_ = 16384, K_ = 4096, NR = 8, KSH = K_ / NR;
constexpr int NKT = K_ / 128;
constexpr unsigned BLK = 16384;
constexpr size_t XI_BYTES = (size_t)(M_ / 128) * NKT * BLK;
constexpr size_t WI_BYTES = (size_t)(N_ / 128) * NKT * BLK;
constexpr size_t BUF_BYTES = XI_BYTES + WI_BYTES;
constexpr size_t CTRL = 1u << 20;
constexpr size_t WS_NEED = CTRL + 2 * BUF_BYTES;
constexpr int NUX = M_ / 256, NUW = N_ / 256, NU = NUX + NUW;
constexpr size_t OFF_TICKET = 0, OFF_TMO = 256, OFF_BUILT = 512, OFF_FLAGS = 4096;
constexpr int GRID = 256, NITEM = NU * 8;
#define STAUX 17
#define L128 1
#define OFL "nt"
#define OST "buffer_store_dwordx4"
#define PWV 2
#define COPY 1
#define PPX 4
constexpr int LDS_BYTES = 9 * (int)BLK;
constexpr unsigned long long DEADLINE = 50000000ull;

struct Params {
    const unsigned char* x;
    const unsigned char* w;
    const float* sx; const float* sw;
    float* out;
    char* ws[NR];
    int rank;
    int mode;
};

#define TS(k) do { } while (0)
#define TS0(k) do { } while (0)
#define PM1
#define QTS
#define BP 7
#define BPRIO 1
#define GRAY 1
#define QM 6

#define PAT 611
__device__ __forceinline__ void item_decode(int it, int lk, int& ub, int& ks, int& pos) {
    int e, r;
    if (it >= PAT && it < PAT + 8) {
        r = it - PAT;
        ub = 32 + 126 + (r & 1); ks = r >> 1; pos = lk ? 1 + (8 - lk) % 7 : 0;
        return;
    }
    if (it >= PAT) it -= 8;
    if (it < 198) {
        e = it < 39 ? 0 : it < 85 ? 1 : it < 138 ? 2 : 3;
        r = it - (it < 39 ? 0 : it < 85 ? 39 : it < 138 ? 85 : 138) - 32;
        if (r < 0) { ub = r + 32; ks = e; pos = lk; return; }
    } else if (it < 590) {
        e = 4 + (it - 198) / 28; r = it - 198 - (e - 4) * 28;
    } else {
        e = it < 611 ? 18 : it < 625 ? 19 : 20;
        r = it - (it < 611 ? 590 : it < 625 ? 611 : 625);
    }
    const int a = e < 3 ? e + 1 : e > 17 ? 21 - e : 4, n0 = e < 3 ? 0 : e - 3;
    const int i = a == 4 ? r >> 2 : a == 3 ? r / 3 : a == 2 ? r >> 1 : r;
    const int j = (lk + 6 + i) % 7;
    ub = 32 + j + 7 * (n0 + r - i * a); ks = e & 3; pos = lk ? i + 1 : 0;
}

__device__ __forceinline__ char* uni(const char* q) {
    const unsigned long long a = (unsigned long long)q;
    return (char*)(((unsigned long long)(unsigned)__builtin_amdgcn_readfirstlane((unsigned)(a >> 32)) << 32) | (unsigned)__builtin_amdgcn_readfirstlane((unsigned)a));
}
#define FENCE do { asm volatile("" ::: "memory"); __builtin_amdgcn_sched_barrier(0); } while (0)

__device__ __forceinline__ void push_load(const Params& p, int it, int lk, int lane, v4u (&r)[16]) {
    int ub, ks, pos; item_decode(it, lk, ub, ks, pos);
    const bool isx = ub < 32;
    const __amdgpu_buffer_rsrc_t rs = __builtin_amdgcn_make_buffer_rsrc(uni(isx ? (const char*)p.x : (const char*)p.w), 0, isx ? M_ * KSH : KSH * N_, 0x00020000);
    const unsigned so = isx ? (unsigned)(ub * 128 * KSH + ks * 128) : (unsigned)(ks * 128 * N_ + (ub - 32) * 128);
    const unsigned s2 = isx ? 16u * KSH : 2u * N_, s1 = isx ? 16u : (unsigned)N_;
    const unsigned vo = isx ? (unsigned)(lane & 15) * KSH + (unsigned)(lane >> 4) * 32u : (unsigned)(lane >> 3) * 16u * N_ + (unsigned)(lane & 7) * 16u;
#pragma unroll
    for (int q = 0; q < 16; ++q)
        r[q] = __builtin_amdgcn_raw_buffer_load_b128(rs, vo, so + (unsigned)(q >> 1) * s2 + (unsigned)(q & 1) * s1, 0);
}

__device__ __forceinline__ void push_phase(const Params& p, LAS unsigned char* lds, unsigned call, int lk, int pw, int npw) {
    const int lane = threadIdx.x & 63, wid = __builtin_amdgcn_readfirstlane(threadIdx.x >> 6);
    LAS unsigned char* myl = lds + wid * BLK;
    char* dbase = uni(p.ws[(unsigned)(p.rank + lk) & 7u]);
    const __amdgpu_buffer_rsrc_t rsD = __builtin_amdgcn_make_buffer_rsrc(dbase + CTRL + (size_t)(call & 1u) * BUF_BYTES, 0, (int)BUF_BYTES, 0x00020000);
    const __amdgpu_buffer_rsrc_t rsG = __builtin_amdgcn_make_buffer_rsrc(dbase + OFF_FLAGS, 0, NU * 256, 0x00020000);
    const unsigned lo = (unsigned)lane * 16u, fdrop = lane ? 0x40000000u : 0u;
    v4u r[16];
    push_load(p, pw, lk, lane, r);
    asm volatile("" : "+v"(r[0]), "+v"(r[1]), "+v"(r[2]), "+v"(r[3]), "+v"(r[4]), "+v"(r[5]), "+v"(r[6]), "+v"(r[7]),
                      "+v"(r[8]), "+v"(r[9]), "+v"(r[10]), "+v"(r[11]), "+v"(r[12]), "+v"(r[13]), "+v"(r[14]), "+v"(r[15]));
    unsigned pfo = 0x40000000u;
    for (int it = pw; it < NITEM; it += npw) {
        int ub, ks, pos; item_decode(it, lk, ub, ks, pos);
        v4u v[16];
        if (ub < 32) {
#pragma unroll
            for (int q = 0; q < 16; ++q) v[q] = r[q];
        } else {
            const int a = lane & 7, b = lane >> 3;
            LAS unsigned char* wl = myl + a * 2048 + (b & 1) * 1024 + (b >> 1) * 256;
#pragma unroll
            for (int wv = 0; wv < 4; ++wv) {
                v4u o[4];
#pragma unroll
                for (int g = 0; g < 4; ++g) {
                    const unsigned r0 = r[4 * g + 0][wv], r1 = r[4 * g + 1][wv], r2 = r[4 * g + 2][wv], r3 = r[4 * g + 3][wv];
                    const unsigned t0 = __builtin_amdgcn_perm(r1, r0, 0x05010400u), t1 = __builtin_amdgcn_perm(r1, r0, 0x07030602u);
                    const unsigned t2 = __builtin_amdgcn_perm(r3, r2, 0x05010400u), t3 = __builtin_amdgcn_perm(r3, r2, 0x07030602u);
                    o[0][g] = __builtin_amdgcn_perm(t2, t0, 0x05040100u);
                    o[1][g] = __builtin_amdgcn_perm(t2, t0, 0x07060302u);
                    o[2][g] = __builtin_amdgcn_perm(t3, t1, 0x05040100u);
                    o[3][g] = __builtin_amdgcn_perm(t3, t1, 0x07060302u);
                }
#pragma unroll
                for (int c = 0; c < 4; ++c) *(LAS v4u*)(wl + (4 * wv + c) * 16) = o[c];
            }
            asm volatile("s_waitcnt lgkmcnt(0)" ::: "memory");
            __builtin_amdgcn_wave_barrier();
#pragma unroll
            for (int q = 0; q < 16; ++q) v[q] = *(LAS v4u*)(myl + q * 1024 + lane * 16);
            asm volatile("s_waitcnt lgkmcnt(0)" ::: "memory");
            __builtin_amdgcn_wave_barrier();
        }
        FENCE;
        push_load(p, it + npw < NITEM ? it + npw : it, lk, lane, r);
        FENCE;
        const unsigned base = (unsigned)(ub * NKT + ks * 8 + pos) * BLK;
#pragma unroll
        for (int q = 0; q < 16; ++q) __builtin_amdgcn_raw_buffer_store_b128(v[q], rsD, lo, base + q * 1024u, STAUX);
        FENCE;
        asm volatile("s_waitcnt vmcnt(33)" ::: "memory");
        __builtin_amdgcn_raw_buffer_store_b32(call, rsG, pfo | fdrop, 0, 17);
        FENCE;
        pfo = (unsigned)((ub >> 1) * 64 + ks * 16 + (ub & 1) * 8 + pos) * 4u;
    }
    asm volatile("s_waitcnt vmcnt(0)" ::: "memory");
    __builtin_amdgcn_raw_buffer_store_b32(call, rsG, pfo | fdrop, 0, 17);
}

__device__ __forceinline__ unsigned long long poll_flags(const Params& p, unsigned call, unsigned fo, LAS unsigned* badw, unsigned long long mk) {
    char* ws = p.ws[p.rank];
    const unsigned* f = (const unsigned*)(ws + OFF_FLAGS + fo);
    unsigned* tmo = (unsigned*)(ws + OFF_TMO);
    const unsigned long long t0 = __builtin_amdgcn_s_memrealtime();
    unsigned long long rd;
    for (;;) {
        const unsigned a = __hip_atomic_load(f, __ATOMIC_RELAXED, __HIP_MEMORY_SCOPE_SYSTEM);
        rd = __ballot(a >= call);
        if ((rd & mk) == mk) break;
        const unsigned tv = (unsigned)__builtin_amdgcn_readfirstlane(__hip_atomic_load(tmo, __ATOMIC_RELAXED, __HIP_MEMORY_SCOPE_AGENT));
        if (tv == call || __builtin_amdgcn_s_memrealtime() - t0 > DEADLINE) {
            if ((threadIdx.x & 63) == 0) { __hip_atomic_store(tmo, call, __ATOMIC_RELAXED, __HIP_MEMORY_SCOPE_AGENT); *badw = 1u; }
            break;
        }
        __builtin_amdgcn_s_sleep(2);
    }
    asm volatile("s_waitcnt vmcnt(0) lgkmcnt(0)" ::: "memory");
    return rd;
}

template <int V> struct IC { static constexpr int v = V; };
constexpr int n_ops(int T, int P) { return P == 0 ? 2 : P == 3 ? 2 : 3; }
constexpr int n_all(int T) { return n_ops(T, 0) + n_ops(T, 1) + n_ops(T, 2) + n_ops(T, 3); }
constexpr int w1(int T) { return 1 + n_all(T + 7) + n_ops(T, 0) + n_ops(T, 1); }
constexpr int w2(int T) { return n_all(T + 7) - n_ops(T + 7, 0) + n_all(T) - n_ops(T, 3); }
constexpr int w3(int T) { return n_ops(T + 7, 3) + n_all(T); }

__device__ __forceinline__ void gemm_phase(const Params& p, LAS unsigned char* lds, LAS unsigned* badw, unsigned call, int ci, int grp, int l0, int lstride, int nstrip) {
    const int tid = threadIdx.x, wid = __builtin_amdgcn_readfirstlane(tid >> 6), lane = tid & 63, rg = wid >> 1, ch = wid & 1;
    char* wsl = p.ws[p.rank];
    const __amdgpu_buffer_rsrc_t rsI = __builtin_amdgcn_make_buffer_rsrc(wsl + CTRL + (size_t)(call & 1u) * BUF_BYTES, 0, (int)BUF_BYTES, 0x00020000);
    const __amdgpu_buffer_rsrc_t rsF = __builtin_amdgcn_make_buffer_rsrc(wsl + OFF_FLAGS, 0, NU * 256, 0x00020000);
    const __amdgpu_buffer_rsrc_t rsO = __builtin_amdgcn_make_buffer_rsrc(p.out, 0, (int)((size_t)M_ * N_ * 4), 0x00020000);
    const unsigned scale = (unsigned)__builtin_amdgcn_readfirstlane(__builtin_bit_cast(int, p.sx[0] * p.sw[0]));
    constexpr unsigned kstep = BLK, hstep = NKT * BLK;
    const unsigned voff = (unsigned)tid * 16u;
    const unsigned ldsw = (unsigned)wid * 1024u;
    const unsigned voffA = (unsigned)(wid >> 2) * hstep + (unsigned)(tid & 255) * 16u, ldswA = (unsigned)(wid >> 2) * 16384u + (unsigned)(wid & 3) * 1024u;
    unsigned abase = (unsigned)(size_t)lds + rg * 8192 + lane * 16, bbase = (unsigned)(size_t)lds + 65536 + ch * 4096 + lane * 16;
    asm volatile("" : "+v"(abase), "+v"(bbase));
    unsigned vq = ((unsigned)(rg * 64) + (lane & 7)) * (N_ * 4u) + (unsigned)(ch * 128) + (lane & 48u) + (lane & 8u) * 8u + 0x80000000u;
    asm volatile("" : "+v"(vq));
#define STG(ldsoff, soff) __builtin_amdgcn_raw_ptr_buffer_load_lds(rsI, (LAS void*)(lds + (ldsoff) + ldsw), 16, voff, (soff), 0, 0)
#define SETA(kk) do { asm volatile("" : "+s"(tbl)); cAk = cA + (((tbl >> (4 * ((kk) & 7))) & 7u) << 14); } while (0)
#define STG_A(b, part, kk) do { asm volatile("" : "+s"(cAk)); __builtin_amdgcn_raw_ptr_buffer_load_lds(rsI, (LAS void*)(lds + (b) * 32768 + (part) * 4096 + ldswA), 16, voffA, \
        cAk + (unsigned)((part) * 4096u + ((kk) & 24) * kstep), 0, 0); } while (0)
#define STG_B(b, f, kk) do { asm volatile("" : "+s"(bB[f])); STG(65536 + (b) * 32768 + (f) * 8192, bB[f] + (unsigned)((kk) * kstep)); } while (0)
#define LDFRAG(base, off) __builtin_shufflevector(*(const LAS v4i*)(size_t)((base) + (off)), *(const LAS v4i*)(size_t)((base) + (off) + 1024), 0, 1, 2, 3, 4, 5, 6, 7)
#define LDA(b) do { _Pragma("unroll") for (int m = 0; m < 4; ++m) At[m] = LDFRAG(abase, (b) * 32768 + m * 2048); } while (0)
#define LDB(b, f) do { _Pragma("unroll") for (int n = 0; n < 2; ++n) Bt[n] = LDFRAG(bbase, (b) * 32768 + (f) * 8192 + n * 2048); } while (0)
#define WAIT_V(n) asm volatile("s_waitcnt vmcnt(%0)" :: "n"(n) : "memory")
#define WAIT_L(n) asm volatile("s_waitcnt lgkmcnt(" #n ")" ::: "memory")
#define BAR __builtin_amdgcn_s_barrier()
#define SCHED __builtin_amdgcn_sched_barrier(0)
    f32x4 acc[4][4][2];
    f32x4 yy[4][2];
#pragma unroll
    for (int m = 0; m < 4; ++m)
#pragma unroll
        for (int n = 0; n < 2; ++n) {
            yy[m][n] = (f32x4){0.f, 0.f, 0.f, 0.f};
#pragma unroll
            for (int a = 0; a < 4; ++a) acc[a][m][n] = (f32x4){0.f, 0.f, 0.f, 0.f};
        }
    v8i At[4], Bt[2];
    const unsigned cA = (unsigned)ci * 2u * hstep;
    unsigned tbl = 0, cAk = cA;
    {
        const int jg = lstride == 7 ? l0 : 0;
#pragma unroll
        for (int t = 1; t < 8; ++t) tbl |= (unsigned)((jg + 8 - t) % 7 + 1) << (4 * t);
    }
    const unsigned sB0 = (unsigned)(32 + l0) * hstep + (unsigned)grp * 8192u, sBs = (unsigned)lstride * hstep;
    unsigned bB[4] = {sB0, sB0, sB0, sB0};
    const unsigned fx = (unsigned)ci * 256u;
    auto fw = [&](int n) { const int l = l0 + n * lstride; return (unsigned)(NUX * 256 + (l >> 1) * 256 + (l & 1) * 32); };
    unsigned fo = (lane >= 16 ? fw(0) + (unsigned)(lane & 7) * 4u : fx + (unsigned)(lane & 8) * 4u + ((tbl >> (4 * (lane & 7))) & 7u) * 4u) + (wid ? 0x40000000u : 0u);
    const unsigned cmpv = wid ? 0u : call;
    constexpr unsigned long long MK = 0x0101010101010101ull;
    const unsigned sO = (unsigned)ci * (256u * N_ * 4u) + (unsigned)grp * 256u;
    unsigned eo = sO;
    float lo = 0.f;
    unsigned long long sc2 = 0;
    unsigned long long rdy = ~0ull;
    unsigned fv = 0, fv2 = 0;
    int tsk = 1;
    int e = 0;
    auto chunk = [&](auto Gc, auto Jc) __attribute__((always_inline)) {
        constexpr int G = decltype(Gc)::v, J = decltype(Jc)::v, m = (J / 5) & 3, r = J % 5;
        if constexpr (J >= 0 && J < 20) {
            if constexpr (r == 0) {
#pragma unroll
                for (int n = 0; n < 2; ++n) {
                    f32x2 h0 = __builtin_shufflevector(acc[G][m][n], acc[G][m][n], 0, 1), h1 = __builtin_shufflevector(acc[G][m][n], acc[G][m][n], 2, 3);
                    asm volatile("v_pk_mul_f32 %0, %0, %1" : "+v"(h0) : "s"(sc2));
                    asm volatile("v_pk_mul_f32 %0, %0, %1" : "+v"(h1) : "s"(sc2));
                    acc[G][m][n] = __builtin_shufflevector(h0, h1, 0, 1, 2, 3);
                }
            } else if constexpr (r <= 2) {
#pragma unroll
                for (int c = 0; c < 4; ++c) asm volatile("v_max_f32 %0, %1, %2" : "=v"(yy[m][r - 1][c]) : "v"(lo), "v"(acc[G][m][r - 1][c]));
            } else if constexpr (r == 3) {
#pragma unroll
                for (int c = 0; c < 4; ++c) asm volatile("v_max_f32_dpp %0, %1, %2 row_ror:8 row_mask:0xf bank_mask:0xc" : "+v"(yy[m][0][c]) : "v"(acc[G][m][1][c]), "v"(lo));
            } else {
#pragma unroll
                for (int c = 0; c < 4; ++c) asm volatile("v_max_f32_dpp %0, %1, %2 row_ror:8 row_mask:0xf bank_mask:0x3" : "+v"(yy[m][1][c]) : "v"(acc[G][m][0][c]), "v"(lo));
            }
        }
    };
    auto mma = [&](auto fc, auto Zc, auto Gc, auto Jc, auto Ac, auto Rc) __attribute__((always_inline)) {
        constexpr int RB = decltype(Rc)::v;
        constexpr int f = decltype(fc)::v, A = decltype(Ac)::v, J0 = A == 3 ? -1 : decltype(Jc)::v;
        constexpr bool Z = decltype(Zc)::v != 0;
        static_assert(A != 2 || J0 < 0, "");
        if (A == 1 || A == 3 || (A == 2 && __builtin_expect(e != 2, 1))) {
            __builtin_amdgcn_s_setprio(1);
#define MF(i) do { constexpr int m = (i) >> 1, n = GRAY ? ((i) & 1) ^ (m & 1) : (i) & 1; \
            if constexpr (Z) asm volatile("v_mfma_f32_16x16x128_f8f6f4 %0, %1, %2, 0 cbsz:1" : "+v"(acc[f][m][n]) : "v"(Bt[n]), "v"(At[m])); \
            else asm volatile("v_mfma_f32_16x16x128_f8f6f4 %0, %1, %2, %0 cbsz:1" : "+v"(acc[f][m][n]) : "v"(Bt[n]), "v"(At[m])); \
            if constexpr (RB >= 0 && ((i) & 1)) { SCHED; At[m] = LDFRAG(abase, RB * 32768 + m * 2048); SCHED; } \
            if constexpr (J0 >= 0 && (i) < 7) chunk(Gc, IC<J0 + (i)>{}); } while (0)
#define MB(i) do { if constexpr (BP == (i) && A != 2) { if (BPRIO) __builtin_amdgcn_s_setprio(2); BAR; } } while (0)
            MF(0); MF(1); MF(2); MF(3); MB(4); MF(4); MB(5); MF(5); MB(6); MF(6); MB(7); MF(7);
#undef MF
            __builtin_amdgcn_s_setprio(0);
        }
        if constexpr (A == 0 && RB >= 0) { LDA(RB); SCHED; }
        if constexpr (BP == 8 || A == 2 || A == 0) BAR;
    };
#define OUT(j) asm volatile(OST " %0, %1, %2, %3 offen " OFL "\n\ts_nop 1" :: "v"(yy[(j) >> 1][(j) & 1]), "v"(vq), "s"(rsO), "s"(eo + (unsigned)((j) * 8) * (N_ * 4u)) : "memory")
#define MMA(P) mma(IC<P>{}, IC<(T == 0 && (P) == F)>{}, IC<(T == 7 ? G : F)>{}, \
                   IC<((T == 7 && (P) > G) ? ((P) - G - 1) * 7 : (T == 0 && (P) < F) ? (3 - F + (P)) * 7 : -1)>{}, IC<ACT(P)>{}, IC<((P) == 3 ? (b ^ 1) : -1)>{})
#define ACT(P) (PRO ? ((P) <= F ? 3 : 0) : 1)
#define LDBA(b, P) do { if constexpr (ACT(P) != 0) LDB(b, P); } while (0)
    auto step = [&](auto Fc, auto Tc, auto Pc) __attribute__((always_inline)) {
        constexpr int F = decltype(Fc)::v, T = decltype(Tc)::v, b = T & 1, G = (F + 1) & 3;
        constexpr bool PRO = decltype(Pc)::v != 0;
        constexpr int k1 = (F * 8 + T + 1) & 31, k2 = (F * 8 + T + 2) & 31;
        if constexpr (T == 0) {
            vq = (vq & 0x7fffffffu) | (e >= 4 ? 0u : 0x80000000u);
            eo = sO + (unsigned)(l0 + (e >= 4 ? e - 4 : 0) * lstride) * 512u;
        }
        if constexpr (T == 0) { TS0(tsk); ++tsk; }
        {
            unsigned long long mk = MK << ((T + 2) & 7);
            if constexpr (T == 7) mk = e + 1 < nstrip ? mk : ~0ull;
            if (__builtin_expect((rdy & mk) != mk, 0)) {
                unsigned fp = fo;
                if constexpr (T == 5) { if ((lane >> 3) == 2 + G) fp = fw(e >= 3 ? e - 3 : 0) + (unsigned)(lane & 7) * 4u; }
                poll_flags(p, call, fp + (T < 6 ? F : G) * 64u, badw, mk);
            }
        }
        if constexpr (T == 4) {
            if ((lane >> 3) == 2 + G && e + 1 < nstrip) fo = fw(e + 1) + (unsigned)(lane & 7) * 4u + (wid ? 0x40000000u : 0u);
        }
        asm volatile("buffer_load_dword %0, %1, %2, 0 offen offset:%3 sc0 sc1" : "=v"(b ? fv2 : fv) : "v"(fo), "s"(rsF), "n"((T < 4 ? F : G) * 64) : "memory");
        if constexpr (T == 7) {
            const bool bad = __builtin_amdgcn_readfirstlane(*badw) != 0u;
            const unsigned s1 = bad ? 0x7fc00000u : scale;
            lo = bad ? __builtin_nanf("") : 0.f; sc2 = ((unsigned long long)s1 << 32) | s1;
            asm volatile("" : "+v"(lo), "+s"(sc2));
        }
#define CONS(P, TT)
#define BAR1(P) BAR
#define BAR2(P)
        LDB(b, 0); STG_B(b ^ 1, 3, k1);
        WAIT_L(0); CONS(3, T - 1); BAR1(0); MMA(0); BAR2(0); SCHED;
        if constexpr (T == 6) {
            bB[G] = e + 1 < nstrip ? sB0 + (unsigned)(e + 1) * sBs : bB[G];
        }
        LDBA(b, 1); STG_B(b, 0, k2); SETA(k2); STG_A(b, 0, k2); STG_A(b, 1, k2);
        WAIT_V(w1(T)); WAIT_L(0); CONS(0, T); BAR1(1); MMA(1); BAR2(1); SCHED;
        LDBA(b, 2); STG_B(b, 1, k2); STG_A(b, 2, k2); STG_A(b, 3, k2);
        WAIT_V(w2(T)); WAIT_L(0); CONS(1, T); BAR1(2); MMA(2); BAR2(2); SCHED;
        LDBA(b, 3); STG_B(b, 2, k2); OUT(T);
        WAIT_V(w3(T));
        asm volatile("" : "+v"(b ? fv : fv2)); rdy = __ballot((b ? fv : fv2) >= cmpv); PM1
        WAIT_L(0); CONS(2, T); BAR1(3); MMA(3); BAR2(3); SCHED;
    };
#define EPOCH(F, PR) \
        step(IC<F>{}, IC<0>{}, IC<PR>{}); step(IC<F>{}, IC<1>{}, IC<PR>{}); step(IC<F>{}, IC<2>{}, IC<PR>{}); step(IC<F>{}, IC<3>{}, IC<PR>{}); \
        step(IC<F>{}, IC<4>{}, IC<PR>{}); step(IC<F>{}, IC<5>{}, IC<PR>{}); step(IC<F>{}, IC<6>{}, IC<PR>{}); step(IC<F>{}, IC<7>{}, IC<PR>{}); ++e;
    if (wid == 0) rdy = poll_flags(p, call, fo, badw, MK * 3u);
    PM1
    asm volatile("" ::: "memory"); BAR; asm volatile("" ::: "memory");
    TS0(tsk); ++tsk;
    asm volatile("buffer_load_dword %0, %1, %2, 0 offen sc0 sc1" : "=v"(fv2) : "v"(fo), "s"(rsF) : "memory");
    STG_B(0, 0, 0); SETA(0); STG_A(0, 0, 0); STG_A(0, 1, 0); STG_A(0, 2, 0); STG_A(0, 3, 0); STG_B(0, 1, 0); STG_B(0, 2, 0); STG_B(0, 3, 0);
    STG_B(1, 0, 1); SETA(1); STG_A(1, 0, 1); STG_A(1, 1, 1); STG_A(1, 2, 1); STG_A(1, 3, 1); STG_B(1, 1, 1); STG_B(1, 2, 1);
    WAIT_V(0); BAR;
    LDA(0); SCHED;
    if (wid >= 4) BAR;
    EPOCH(0, 1) EPOCH(1, 1)
    while (e < nstrip) { EPOCH(2, 0) EPOCH(3, 0) EPOCH(0, 0) EPOCH(1, 0) }
    unsigned bbase0 = bbase - 65536u;
    asm volatile("" : "+v"(bbase0));
    const unsigned rg4 = (unsigned)rg * 4u;
#define TDA(pc, part, kk) do { SETA(kk); STG((pc) * 8192, cAk + (unsigned)(((part) >> 1) * hstep + ((part) & 1) * 8192u + ((kk) & 24) * kstep)); } while (0)
#define TDB(pc, f, kk) do { asm volatile("" : "+s"(bB[f])); STG((pc) * 8192, bB[f] + (unsigned)(TK(kk) * kstep)); } while (0)
#define TRA(pk) do { const unsigned ab_ = abase + (((((unsigned)(pk) >> rg4) & 15u) - (unsigned)rg) << 13); \
        _Pragma("unroll") for (int m = 0; m < 4; ++m) At[m] = LDFRAG(ab_, m * 2048); } while (0)
#define TRB(pc) do { _Pragma("unroll") for (int n = 0; n < 2; ++n) Bt[n] = LDFRAG((pc) >= 8 ? bbase : bbase0, ((pc) & 7) * 8192 + n * 2048); } while (0)
#define TMF(f, i) asm volatile("v_mfma_f32_16x16x128_f8f6f4 %0, %1, %2, %0 cbsz:1" : "+v"(acc[f][(i) >> 1][(i) & 1]) : "v"(Bt[(i) & 1]), "v"(At[(i) >> 1]))
#define TCH(G, J) chunk(IC<G>{}, IC<J>{})
#define PRIO(n) __builtin_amdgcn_s_setprio(n)
#define TEP(i) do { vq = (vq & 0x7fffffffu) | (nstrip + (i) >= 4 ? 0u : 0x80000000u); \
        eo = sO + (unsigned)(l0 + (nstrip + (i) >= 4 ? nstrip + (i) - 4 : 0) * lstride) * 512u; } while (0)
#define TFL(i) asm volatile("buffer_load_dword %0, %1, %2, 0 offen offset:%3 sc0 sc1" : "=v"((i) ? fv2 : fv) : "v"(fo), "s"(rsF), "n"((i) ? 0 : 192) : "memory")
#define TRDY do { asm volatile("" : "+v"(fv), "+v"(fv2)); rdy = __ballot(fv >= cmpv && fv2 >= cmpv); PM1 } while (0)
#define TPOLL do { if (rdy != ~0ull) { poll_flags(p, call, fo + 192u, badw, ~0ull); poll_flags(p, call, fo, badw, ~0ull); } } while (0)
#define TBAD do { const bool bad = __builtin_amdgcn_readfirstlane(*badw) != 0u; const unsigned s1 = bad ? 0x7fc00000u : scale; \
        lo = bad ? __builtin_nanf("") : 0.f; sc2 = ((unsigned long long)s1 << 32) | s1; asm volatile("" : "+v"(lo), "+s"(sc2)); } while (0)
#define TTS
#define TK(kk) (kk)
    TEP(0); TTS; TRB(8); SCHED; LDA(0); TFL(0); TDB(15, 3, 17); TDB(10, 0, 18); WAIT_L(8); BAR; WAIT_L(0);
    PRIO(1); TMF(0, 0); TCH(2, 7); TMF(0, 1); TCH(2, 8); TMF(0, 2); TCH(2, 9); TMF(0, 3); TCH(2, 10); TMF(0, 4); TCH(2, 11); TMF(0, 5); TCH(2, 12); TMF(0, 6); TCH(2, 13); TMF(0, 7); PRIO(0); BAR; SCHED;
    TRB(9); TDA(0, 0, 18); TDA(1, 1, 18); TDA(14, 2, 18); WAIT_V(14); WAIT_L(0); BAR;
    PRIO(1); TMF(1, 0); TCH(2, 14); TMF(1, 1); TCH(2, 15); TMF(1, 2); TCH(2, 16); TMF(1, 3); TCH(2, 17); TMF(1, 4); TCH(2, 18); TMF(1, 5); TCH(2, 19); TMF(1, 6); TMF(1, 7); PRIO(0); BAR; SCHED;
    TRB(11); TDA(3, 3, 18); TDB(9, 1, 18); OUT(0); WAIT_V(14); WAIT_L(0); BAR;
    PRIO(1); TMF(3, 0); TMF(3, 1); TMF(3, 2); TMF(3, 3); TMF(3, 4); TMF(3, 5); TMF(3, 6); TMF(3, 7); PRIO(0); WAIT_V(11); BAR; SCHED;
    TTS; TRB(12); SCHED; LDA(1); TFL(1); TDB(11, 3, 18); TDB(8, 0, 19); WAIT_L(8); BAR; WAIT_L(0);
    PRIO(1); TMF(0, 0); TMF(0, 1); TMF(0, 2); TMF(0, 3); TMF(0, 4); TMF(0, 5); TMF(0, 6); TMF(0, 7); PRIO(0); BAR; SCHED;
    TRB(13); TDA(4, 0, 19); TDA(5, 1, 19); TDA(2, 2, 19); WAIT_V(13); WAIT_L(0); BAR;
    PRIO(1); TMF(1, 0); TMF(1, 1); TMF(1, 2); TMF(1, 3); TMF(1, 4); TMF(1, 5); TMF(1, 6); TMF(1, 7); PRIO(0); BAR; SCHED;
    TRB(15); TDA(7, 3, 19); TDB(13, 1, 19); OUT(1); WAIT_V(13); WAIT_L(0); BAR;
    PRIO(1); TMF(3, 0); TMF(3, 1); TMF(3, 2); TMF(3, 3); TMF(3, 4); TMF(3, 5); TMF(3, 6); TMF(3, 7); PRIO(0); WAIT_V(11); BAR; SCHED;
    TTS; TRB(10); SCHED; TRA(0x3e10); TDB(15, 3, 19); TDB(12, 0, 20); TDA(6, 0, 20); WAIT_V(13); WAIT_L(8); BAR; WAIT_L(0);
    PRIO(1); TMF(0, 0); TMF(0, 1); TMF(0, 2); TMF(0, 3); TMF(0, 4); TMF(0, 5); TMF(0, 6); TMF(0, 7); PRIO(0); BAR; SCHED;
    TRB(9); TDA(1, 1, 20); TDA(0, 2, 20); TDA(10, 3, 20); WAIT_V(13); WAIT_L(0); BAR;
    PRIO(1); TMF(1, 0); TMF(1, 1); TMF(1, 2); TMF(1, 3); TMF(1, 4); TMF(1, 5); TMF(1, 6); TMF(1, 7); PRIO(0); BAR; SCHED;
    TRB(11); TDB(9, 1, 20); TDB(3, 3, 20); OUT(2); WAIT_V(13); TRDY; WAIT_L(0); BAR;
    PRIO(1); TMF(3, 0); TMF(3, 1); TMF(3, 2); TMF(3, 3); TMF(3, 4); TMF(3, 5); TMF(3, 6); TMF(3, 7); PRIO(0); WAIT_V(11); BAR; SCHED;
    TTS; TPOLL; TRB(8); SCHED; TRA(0x7254); TDB(14, 0, 21); TDA(11, 0, 21); WAIT_V(12); WAIT_L(8); BAR; WAIT_L(0);
    PRIO(1); TMF(0, 0); TMF(0, 1); TMF(0, 2); TMF(0, 3); TMF(0, 4); TMF(0, 5); TMF(0, 6); TMF(0, 7); PRIO(0); BAR; SCHED;
    TRB(13); TDA(5, 1, 21); TDA(4, 2, 21); TDA(8, 3, 21); WAIT_V(13); WAIT_L(0); BAR;
    PRIO(1); TMF(1, 0); TMF(1, 1); TMF(1, 2); TMF(1, 3); TMF(1, 4); TMF(1, 5); TMF(1, 6); TMF(1, 7); PRIO(0); BAR; SCHED;
    TRB(15); TDB(13, 1, 21); TDB(2, 3, 21); OUT(3); WAIT_V(13); WAIT_L(0); BAR;
    PRIO(1); TMF(3, 0); TMF(3, 1); TMF(3, 2); TMF(3, 3); TMF(3, 4); TMF(3, 5); TMF(3, 6); TMF(3, 7); PRIO(0); WAIT_V(11); BAR; SCHED;
    TTS; TRB(12); SCHED; TRA(0xa016); TDB(7, 0, 22); TDA(15, 0, 22); WAIT_V(12); WAIT_L(8); BAR; WAIT_L(0);
    PRIO(1); TMF(0, 0); TMF(0, 1); TMF(0, 2); TMF(0, 3); TMF(0, 4); TMF(0, 5); TMF(0, 6); TMF(0, 7); PRIO(0); BAR; SCHED;
    TRB(9); TDA(1, 1, 22); TDA(6, 2, 22); TDA(12, 3, 22); WAIT_V(14); WAIT_L(0); BAR;
    PRIO(1); TMF(1, 0); TMF(1, 1); TMF(1, 2); TMF(1, 3); TMF(1, 4); TMF(1, 5); TMF(1, 6); TMF(1, 7); PRIO(0); BAR; SCHED;
    TRB(3); TDB(9, 1, 22); TDB(0, 3, 22); OUT(4); WAIT_V(13); WAIT_L(0); BAR;
    PRIO(1); TMF(3, 0); TMF(3, 1); TMF(3, 2); TMF(3, 3); TMF(3, 4); TMF(3, 5); TMF(3, 6); TMF(3, 7); PRIO(0); WAIT_V(11); BAR; SCHED;
    TTS; TRB(14); SCHED; TRA(0x845b); TDB(10, 0, 23); TDA(3, 0, 23); WAIT_V(12); WAIT_L(8); BAR; WAIT_L(0);
    PRIO(1); TMF(0, 0); TMF(0, 1); TMF(0, 2); TMF(0, 3); TMF(0, 4); TMF(0, 5); TMF(0, 6); TMF(0, 7); PRIO(0); BAR; SCHED;
    TRB(13); TDA(5, 1, 23); TDA(11, 2, 23); TDA(14, 3, 23); WAIT_V(14); WAIT_L(0); BAR;
    PRIO(1); TMF(1, 0); TMF(1, 1); TMF(1, 2); TMF(1, 3); TMF(1, 4); TMF(1, 5); TMF(1, 6); TMF(1, 7); PRIO(0); BAR; SCHED;
    TRB(2); TDB(13, 1, 23); TDB(4, 3, 23); OUT(5); WAIT_V(13); WAIT_L(0); BAR;
    PRIO(1); TMF(3, 0); TMF(3, 1); TMF(3, 2); TMF(3, 3); TMF(3, 4); TMF(3, 5); TMF(3, 6); TMF(3, 7); PRIO(0); WAIT_V(11); BAR; SCHED;
    TTS; TRB(7); SCHED; TRA(0xc61f); TDB(8, 0, 24); TDA(2, 0, 24); WAIT_V(12); WAIT_L(8); BAR; WAIT_L(0);
    PRIO(1); TMF(0, 0); TMF(0, 1); TMF(0, 2); TMF(0, 3); TMF(0, 4); TMF(0, 5); TMF(0, 6); TMF(0, 7); PRIO(0); BAR; SCHED;
    TRB(9); TDA(1, 1, 24); TDA(7, 2, 24); TDA(15, 3, 24); WAIT_V(14); WAIT_L(0); BAR;
    PRIO(1); TMF(1, 0); TMF(1, 1); TMF(1, 2); TMF(1, 3); TMF(1, 4); TMF(1, 5); TMF(1, 6); TMF(1, 7); PRIO(0); BAR; SCHED;
    TRB(0); TDB(9, 1, 24); TDB(12, 0, 25); OUT(6); WAIT_V(13); WAIT_L(0); BAR;
    PRIO(1); TMF(3, 0); TMF(3, 1); TMF(3, 2); TMF(3, 3); TMF(3, 4); TMF(3, 5); TMF(3, 6); TMF(3, 7); PRIO(0); WAIT_V(11); BAR; SCHED;
    TTS; TBAD; TRB(10); SCHED; TRA(0xeb53); TDA(6, 0, 25); TDA(0, 1, 25); WAIT_V(12); WAIT_L(8); BAR; WAIT_L(0);
    PRIO(1); TMF(0, 0); TMF(0, 1); TMF(0, 2); TMF(0, 3); TMF(0, 4); TMF(0, 5); TMF(0, 6); TMF(0, 7); PRIO(0); BAR; SCHED;
    TRB(13); TDA(3, 2, 25); TDA(5, 3, 25); TDB(10, 1, 25); WAIT_V(14); WAIT_L(0); BAR;
    PRIO(1); TMF(1, 0); TMF(1, 1); TMF(1, 2); TMF(1, 3); TMF(1, 4); TMF(1, 5); TMF(1, 6); TMF(1, 7); PRIO(0); BAR; SCHED;
    TRB(4); TDB(11, 0, 26); TDA(14, 0, 26); OUT(7); WAIT_V(13); WAIT_L(0); BAR;
    PRIO(1); TMF(3, 0); TMF(3, 1); TMF(3, 2); TMF(3, 3); TMF(3, 4); TMF(3, 5); TMF(3, 6); TMF(3, 7); PRIO(0); WAIT_V(11); BAR; SCHED;
    TEP(1); TTS; TRB(8); SCHED; TRA(0xf712); TDA(13, 1, 26); TDA(4, 2, 26); WAIT_V(12); WAIT_L(8); BAR; WAIT_L(0);
    PRIO(1); TMF(0, 0); TCH(3, 0); TCH(3, 1); TMF(0, 1); TCH(3, 2); TCH(3, 3); TMF(0, 2); TCH(3, 4); TMF(0, 3); TCH(3, 5); TMF(0, 4); TCH(3, 6); TMF(0, 5); TCH(3, 7); TMF(0, 6); TCH(3, 8); TMF(0, 7); TCH(3, 9); PRIO(0); BAR; SCHED;
    TRB(9); TDA(1, 3, 26); TDB(2, 1, 26); TDB(8, 0, 27); OUT(0); WAIT_V(12); WAIT_L(0); BAR;
    PRIO(1); TMF(1, 0); TCH(3, 10); TCH(3, 11); TMF(1, 1); TCH(3, 12); TCH(3, 13); TMF(1, 2); TCH(3, 14); TMF(1, 3); TCH(3, 15); TMF(1, 4); TCH(3, 16); TMF(1, 5); TCH(3, 17); TMF(1, 6); TCH(3, 18); TMF(1, 7); TCH(3, 19); PRIO(0); WAIT_V(10); BAR; SCHED;
    TTS; TRB(12); SCHED; TRA(0x5306); TDA(7, 0, 27); TDA(15, 1, 27); TDA(9, 2, 27); WAIT_V(12); WAIT_L(8); BAR; WAIT_L(0);
    PRIO(1); TMF(0, 0); TMF(0, 1); TMF(0, 2); TMF(0, 3); TMF(0, 4); TMF(0, 5); TMF(0, 6); TMF(0, 7); PRIO(0); BAR; SCHED;
    TRB(10); TDA(0, 3, 27); TDB(6, 1, 27); TDB(12, 0, 28); OUT(1); WAIT_V(12); WAIT_L(0); BAR;
    PRIO(1); TMF(1, 0); TMF(1, 1); TMF(1, 2); TMF(1, 3); TMF(1, 4); TMF(1, 5); TMF(1, 6); TMF(1, 7); PRIO(0); WAIT_V(10); BAR; SCHED;
    TTS; TRB(11); SCHED; TRA(0x14de); TDA(3, 0, 28); TDA(5, 1, 28); TDA(10, 2, 28); WAIT_V(12); WAIT_L(8); BAR; WAIT_L(0);
    PRIO(1); TMF(0, 0); TMF(0, 1); TMF(0, 2); TMF(0, 3); TMF(0, 4); TMF(0, 5); TMF(0, 6); TMF(0, 7); PRIO(0); BAR; SCHED;
    TRB(2); TDA(11, 3, 28); TDB(13, 1, 28); TDB(14, 0, 29); OUT(2); WAIT_V(12); WAIT_L(0); BAR;
    PRIO(1); TMF(1, 0); TMF(1, 1); TMF(1, 2); TMF(1, 3); TMF(1, 4); TMF(1, 5); TMF(1, 6); TMF(1, 7); PRIO(0); WAIT_V(10); BAR; SCHED;
    TTS; TRB(8); SCHED; TRA(0x09f7); TDA(4, 0, 29); TDA(1, 1, 29); TDA(2, 2, 29); WAIT_V(12); WAIT_L(8); BAR; WAIT_L(0);
    PRIO(1); TMF(0, 0); TMF(0, 1); TMF(0, 2); TMF(0, 3); TMF(0, 4); TMF(0, 5); TMF(0, 6); TMF(0, 7); PRIO(0); BAR; SCHED;
    TRB(6); TDA(7, 3, 29); TDB(8, 1, 29); TDB(15, 0, 30); OUT(3); WAIT_V(12); WAIT_L(0); BAR;
    PRIO(1); TMF(1, 0); TMF(1, 1); TMF(1, 2); TMF(1, 3); TMF(1, 4); TMF(1, 5); TMF(1, 6); TMF(1, 7); PRIO(0); WAIT_V(10); BAR; SCHED;
    TTS; TRB(12); SCHED; TRA(0xba53); TDA(0, 0, 30); TDA(9, 1, 30); TDA(6, 2, 30); WAIT_V(12); WAIT_L(8); BAR; WAIT_L(0);
    PRIO(1); TMF(0, 0); TMF(0, 1); TMF(0, 2); TMF(0, 3); TMF(0, 4); TMF(0, 5); TMF(0, 6); TMF(0, 7); PRIO(0); BAR; SCHED;
    TRB(13); TDA(3, 3, 30); TDB(5, 1, 30); TDB(12, 0, 31); OUT(4); WAIT_V(12); WAIT_L(0); BAR;
    PRIO(1); TMF(1, 0); TMF(1, 1); TMF(1, 2); TMF(1, 3); TMF(1, 4); TMF(1, 5); TMF(1, 6); TMF(1, 7); PRIO(0); WAIT_V(10); BAR; SCHED;
    TTS; TRB(14); SCHED; TRA(0x7214); TDA(10, 0, 31); TDA(11, 1, 31); TDA(13, 2, 31); WAIT_V(12); WAIT_L(8); BAR; WAIT_L(0);
    PRIO(1); TMF(0, 0); TMF(0, 1); TMF(0, 2); TMF(0, 3); TMF(0, 4); TMF(0, 5); TMF(0, 6); TMF(0, 7); PRIO(0); BAR; SCHED;
    TRB(8); TDA(1, 3, 31); TDB(4, 1, 31); TDB(14, 1, 0); OUT(5); WAIT_V(12); WAIT_L(0); BAR;
    PRIO(1); TMF(1, 0); TMF(1, 1); TMF(1, 2); TMF(1, 3); TMF(1, 4); TMF(1, 5); TMF(1, 6); TMF(1, 7); PRIO(0); WAIT_V(10); BAR; SCHED;
    TTS; TRB(15); SCHED; TRA(0x3690); TDA(2, 0, 0); TDA(7, 1, 0); TDA(8, 2, 0); WAIT_V(12); WAIT_L(8); BAR; WAIT_L(0);
    PRIO(1); TMF(0, 0); TMF(0, 1); TMF(0, 2); TMF(0, 3); TMF(0, 4); TMF(0, 5); TMF(0, 6); TMF(0, 7); PRIO(0); BAR; SCHED;
    TRB(5); TDA(0, 3, 0); TDB(9, 1, 1); TDA(15, 0, 1); OUT(6); WAIT_V(12); WAIT_L(0); BAR;
    PRIO(1); TMF(1, 0); TMF(1, 1); TMF(1, 2); TMF(1, 3); TMF(1, 4); TMF(1, 5); TMF(1, 6); TMF(1, 7); PRIO(0); WAIT_V(10); BAR; SCHED;
    TTS; TRB(12); SCHED; TRA(0x1dba); TDA(5, 1, 1); TDA(6, 2, 1); TDA(3, 3, 1); WAIT_V(12); WAIT_L(8); BAR; WAIT_L(0);
    PRIO(1); TMF(0, 0); TMF(0, 1); TMF(0, 2); TMF(0, 3); TMF(0, 4); TMF(0, 5); TMF(0, 6); TMF(0, 7); PRIO(0); BAR; SCHED;
    TRB(4); TDB(10, 1, 2); TDA(11, 0, 2); TDA(12, 1, 2); OUT(7); WAIT_V(12); WAIT_L(0); BAR;
    PRIO(1); TMF(1, 0); TCH(0, 0); TCH(0, 1); TMF(1, 1); TCH(0, 2); TCH(0, 3); TMF(1, 2); TCH(0, 4); TMF(1, 3); TCH(0, 5); TMF(1, 4); TCH(0, 6); TMF(1, 5); TCH(0, 7); TMF(1, 6); TCH(0, 8); TMF(1, 7); TCH(0, 9); PRIO(0); WAIT_V(10); BAR; SCHED;
    TEP(2); TTS; TRB(14); SCHED; TRA(0x0872); TDA(1, 2, 2); TDA(13, 3, 2); TDB(4, 1, 3); OUT(0); WAIT_V(10); WAIT_L(8); BAR; WAIT_L(0);
    PRIO(1); TMF(1, 0); TCH(0, 10); TCH(0, 11); TMF(1, 1); TCH(0, 12); TCH(0, 13); TMF(1, 2); TCH(0, 14); TMF(1, 3); TCH(0, 15); TMF(1, 4); TCH(0, 16); TMF(1, 5); TCH(0, 17); TMF(1, 6); TCH(0, 18); TMF(1, 7); TCH(0, 19); PRIO(0); WAIT_V(8); BAR; SCHED;
    TTS; TRB(9); SCHED; TRA(0x365f); TDA(2, 0, 3); TDA(7, 1, 3); TDA(14, 2, 3); OUT(1); WAIT_V(9); WAIT_L(8); BAR; WAIT_L(0);
    PRIO(1); TMF(1, 0); TMF(1, 1); SCHED; TDA(0, 3, 3); SCHED; TMF(1, 2); TMF(1, 3); SCHED; TDB(8, 1, 4); SCHED; TMF(1, 4); TMF(1, 5); TMF(1, 6); TMF(1, 7); PRIO(0); WAIT_V(8); BAR; SCHED;
    TTS; TRB(10); SCHED; TRA(0xd1cb); TDA(5, 0, 4); TDA(9, 1, 4); TDA(15, 2, 4); OUT(2); WAIT_V(8); WAIT_L(8); BAR; WAIT_L(0);
    PRIO(1); TMF(1, 0); TMF(1, 1); SCHED; TDA(3, 3, 4); SCHED; TMF(1, 2); TMF(1, 3); SCHED; TDB(6, 1, 5); SCHED; TMF(1, 4); TMF(1, 5); TMF(1, 6); TMF(1, 7); PRIO(0); WAIT_V(7); BAR; SCHED;
    TTS; TRB(4); SCHED; TRA(0x0e72); TDA(10, 0, 5); TDA(11, 1, 5); TDA(12, 2, 5); OUT(3); WAIT_V(8); WAIT_L(8); BAR; WAIT_L(0);
    PRIO(1); TMF(1, 0); TMF(1, 1); SCHED; TDA(1, 3, 5); SCHED; TMF(1, 2); TMF(1, 3); SCHED; TDB(13, 1, 6); SCHED; TMF(1, 4); TMF(1, 5); TMF(1, 6); TMF(1, 7); PRIO(0); WAIT_V(7); BAR; SCHED;
    TTS; TRB(8); SCHED; TRA(0x3f95); TDA(2, 0, 6); TDA(4, 1, 6); TDA(7, 2, 6); OUT(4); WAIT_V(8); WAIT_L(8); BAR; WAIT_L(0);
    PRIO(1); TMF(1, 0); TMF(1, 1); SCHED; TDA(0, 3, 6); SCHED; TMF(1, 2); TMF(1, 3); SCHED; TDB(14, 1, 7); SCHED; TMF(1, 4); TMF(1, 5); TMF(1, 6); TMF(1, 7); PRIO(0); WAIT_V(7); BAR; SCHED;
    TTS; TRB(6); SCHED; TRA(0x1cba); TDA(5, 0, 7); TDA(8, 1, 7); TDA(9, 2, 7); OUT(5); WAIT_V(8); WAIT_L(8); BAR; WAIT_L(0);
    PRIO(1); TMF(1, 0); TMF(1, 1); SCHED; TDA(3, 3, 7); SCHED; TMF(1, 2); TMF(1, 3); TMF(1, 4); TMF(1, 5); TMF(1, 6); TMF(1, 7); PRIO(0); WAIT_V(6); BAR; SCHED;
    TTS; TRB(13); SCHED; TRA(0x0742); OUT(6); WAIT_V(4); WAIT_L(8); BAR; WAIT_L(0);
    PRIO(1); TMF(1, 0); TMF(1, 1); TMF(1, 2); TMF(1, 3); TMF(1, 4); TMF(1, 5); TMF(1, 6); TMF(1, 7); PRIO(0); WAIT_V(1); BAR; SCHED;
    TTS; TRB(14); SCHED; TRA(0x3985); OUT(7); WAIT_L(8); BAR; WAIT_L(0);
    PRIO(1); TMF(1, 0); TMF(1, 1); TMF(1, 2); TMF(1, 3); TMF(1, 4); TMF(1, 5); TMF(1, 6); TMF(1, 7); PRIO(0); SCHED;
    TEP(3);
    asm volatile("s_nop 15\n\ts_nop 15" ::: "memory");
    TCH(1, 0); TCH(1, 1); TCH(1, 2); TCH(1, 3); TCH(1, 4); OUT(0); OUT(1);
    TCH(1, 5); TCH(1, 6); TCH(1, 7); TCH(1, 8); TCH(1, 9); OUT(2); OUT(3);
    TCH(1, 10); TCH(1, 11); TCH(1, 12); TCH(1, 13); TCH(1, 14); OUT(4); OUT(5);
    TCH(1, 15); TCH(1, 16); TCH(1, 17); TCH(1, 18); TCH(1, 19); OUT(6); OUT(7);
    TS0(tsk); ++tsk;
    WAIT_V(0);
    if (wid < 4) BAR;
#undef STG
#undef STG_A
#undef STG_B
#undef LDFRAG
#undef LDA
#undef LDB
#undef MMA
#undef WAIT_V
#undef WAIT_L
#undef BAR
#undef SCHED
#undef OUT
#undef EPOCH
}

__global__ __launch_bounds__(512, 2) void fused_kernel(const Params p) {
    extern __shared__ __attribute__((aligned(16))) unsigned char shm[];
    LAS unsigned char* lds = (LAS unsigned char*)shm;
    LAS unsigned* ctl = (LAS unsigned*)(lds + LDS_BYTES);
    if (threadIdx.x == 0) {
        const unsigned ticket = __hip_atomic_fetch_add((unsigned*)(p.ws[p.rank] + OFF_TICKET), 1u, __ATOMIC_RELAXED, __HIP_MEMORY_SCOPE_AGENT);
        ctl[0] = ticket / (unsigned)GRID + 1u; ctl[1] = 0u;
    }
    __syncthreads();
    const unsigned call = (unsigned)__builtin_amdgcn_readfirstlane(ctl[0]);
    const int bid = blockIdx.x;
    TS0(0);
    const int xcd = bid & 7, slot = bid >> 3, wid = __builtin_amdgcn_readfirstlane(threadIdx.x >> 6);
    const bool pusher = slot < 4;
    if (pusher) {
        if (wid < PWV) push_phase(p, lds, call, xcd, slot * PWV + wid, 4 * PWV);
        asm volatile("s_waitcnt vmcnt(0)" ::: "memory");
        __syncthreads();
    }
    else {
        const unsigned g = (unsigned)((slot - 4) * 8 + xcd) * 512u + threadIdx.x;
        const __amdgpu_buffer_rsrc_t rx = __builtin_amdgcn_make_buffer_rsrc(uni((const char*)p.x), 0, M_ * KSH, 0x00020000);
        const __amdgpu_buffer_rsrc_t rw = __builtin_amdgcn_make_buffer_rsrc(uni((const char*)p.w), 0, KSH * N_, 0x00020000);
        v4u t[7];
#pragma unroll
        for (int i = 0; i < 5; ++i) t[i] = __builtin_amdgcn_raw_buffer_load_b128(rw, g * 16u, (unsigned)i * (224u * 8192u), 0);
#pragma unroll
        for (int i = 0; i < 2; ++i) t[5 + i] = __builtin_amdgcn_raw_buffer_load_b128(rx, g * 16u, (unsigned)i * (224u * 8192u), 0);
        asm volatile("" :: "v"(t[0]), "v"(t[1]), "v"(t[2]), "v"(t[3]), "v"(t[4]), "v"(t[5]), "v"(t[6]));
    }
    TS0(31);
    gemm_phase(p, lds, ctl + 1, call, (xcd & 3) * 4 + (slot & 3), xcd >> 2, pusher ? 126 : (slot >> 2) - 1, pusher ? 1 : 7, pusher ? 2 : 18);
}

}

extern "C" void kernel_launch(int rank, int nranks, ncclComm_t comm,
                              hipStream_t stream,
                              void* const* d_in, const int* in_sizes, int n_in,
                              void* d_out, int out_size,
                              const int* in_dtypes, int out_dtype,
                              void* const* d_out_all,
                              void* const* d_ws_all, size_t ws_bytes) {
    using namespace fk;
    if (nranks != NR || ws_bytes < WS_NEED || (unsigned)rank >= (unsigned)NR) { fprintf(stderr, "unexpected configuration\n"); return; }
    static bool attr_done = false;
    if (!attr_done) {
        HIP_CHECK(hipFuncSetAttribute((const void*)fused_kernel, hipFuncAttributeMaxDynamicSharedMemorySize, LDS_BYTES + 64));
        attr_done = true;
    }
    Params p;
    p.x = (const unsigned char*)d_in[0]; p.w = (const unsigned char*)d_in[1];
    p.sx = (const float*)d_in[2]; p.sw = (const float*)d_in[3];
    p.out = (float*)d_out; p.rank = rank; p.mode = 0;
    for (int r = 0; r < NR; ++r) p.ws[r] = (char*)d_ws_all[r];
    hipLaunchKernelGGL(fused_kernel, dim3(GRID), dim3(512), LDS_BYTES + 64, stream, p);
    HIP_CHECK(hipGetLastError());
}
```
